```python
import numpy as np

kernel_name = "gpu_gemm_a2a_m4096_k4096_n16384_e4m3e5m2_dqsilu"


ABSMAX_RTOL = 0.05

COMPUTE_DTYPE = "float32"
EPILOGUE = "dequant_silu"
IN_QUANT = None
OUT_QUANT = None
IN_FP8 = "e4m3e5m2"
IN_DTYPES = {"x": "e4m3", "w_mat": "e5m2",
             "scale_x": "float32", "scale_w": "float32"}
WORLD_SIZE = 8

def _snap_fp8(x, mant_bits, max_abs, min_normal):
    a = np.abs(x.astype(np.float64))
    _m, _e = np.frexp(a)
    q = np.where(a >= min_normal, np.ldexp(1.0, _e - 1 - mant_bits),
                 min_normal * 2.0 ** -mant_bits)
    snapped = np.minimum(np.round(a / q) * q, max_abs)
    return (np.sign(x) * snapped).astype(np.float32)

def setup_inputs():

    x = _snap_fp8(np.random.randn(4096, 4096).astype(np.float32), 3, 448.0, 2.0 ** -6)
    w_mat = _snap_fp8(np.random.randn(4096, 16384).astype(np.float32), 2, 57344.0, 2.0 ** -14)
    scale_x = (0.02 * (1.0 + np.random.rand(1))).astype(np.float32)
    scale_w = (0.02 * (1.0 + np.random.rand(1))).astype(np.float32)
    return {"x": x, "w_mat": w_mat, "scale_x": scale_x, "scale_w": scale_w}

def reference(x, w_mat, scale_x, scale_w):
    acc = (x @ w_mat).astype(np.float32)
    y = (acc * (scale_x * scale_w)).astype(np.float32)
    return (y / (1.0 + np.exp(-np.clip(y, -60.0, 60.0)))).astype(np.float32)
```

```cpp
#include <hip/hip_runtime.h>
#include <rccl/rccl.h>
#include <cstdio>
#include <cstdint>

#define HIP_CHECK(expr) do { hipError_t _e = (expr); if (_e != hipSuccess) { \
    fprintf(stderr, "HIP error %s at %s:%d\n", hipGetErrorString(_e), __FILE__, __LINE__); \
    return; } } while (0)

namespace {

#define LAS __attribute__((address_space(3)))
typedef int v2i __attribute__((ext_vector_type(2)));
typedef int v4i __attribute__((ext_vector_type(4)));
typedef int v8i __attribute__((ext_vector_type(8)));
typedef unsigned u32x4 __attribute__((ext_vector_type(4)));
typedef float v4f __attribute__((ext_vector_type(4)));

constexpr int NR = 8;
constexpr int MT = 4096, KT = 4096, NT = 16384;
constexpr int MS = MT / NR;
constexpr int NS = NT / NR;
constexpr int BM = 128, BN = 256, BK = 128;
constexpr int NSTEP = KT / BK;
constexpr int NRB = MS / BM;
constexpr int NCB = NS / BN;
constexpr int NWG = (MT / BM) * NCB;
constexpr int A_BYTES = BM * BK;
constexpr int B_BYTES = BK * BN;
constexpr int STAGE_BYTES = A_BYTES + B_BYTES;
constexpr int NSTAGE = 3;
constexpr int LDS_BYTES = NSTAGE * STAGE_BYTES + 64;

constexpr size_t WS_CALL = 0;
constexpr size_t WS_TMO = WS_CALL + (size_t)NWG * 128;
constexpr size_t WS_FLAG = WS_TMO + 128;
constexpr size_t WS_STAGE = 1u << 20;
constexpr size_t IMG_RB = (size_t)NSTEP * A_BYTES;
constexpr size_t IMG_SRC = IMG_RB * NRB;
constexpr size_t IMG_SLOT = IMG_SRC * NR;
constexpr size_t WS_TOTAL = WS_STAGE + 2 * IMG_SLOT;
static_assert(WS_FLAG + (size_t)NR * NRB * 128 <= WS_STAGE, "control block");

constexpr unsigned long long DEADLINE_TICKS = 40000000ull;
constexpr int AUX_SYS = 17;

struct Params {
  const unsigned char* x;
  const unsigned char* w;
  const float* sx;
  const float* sw;
  float* out;
  unsigned char* ws[NR];
  int rank;
};

#define WAIT_V(n) asm volatile("s_waitcnt vmcnt(" #n ")" ::: "memory")
#define BAR() do { asm volatile("" ::: "memory"); __builtin_amdgcn_s_barrier(); asm volatile("" ::: "memory"); } while (0)

__device__ __forceinline__ const void* uni_ptr(const void* p) {
  const unsigned long long v = (unsigned long long)p;
  const unsigned lo = __builtin_amdgcn_readfirstlane((unsigned)v);
  const unsigned hi = __builtin_amdgcn_readfirstlane((unsigned)(v >> 32));
  return (const void*)(((unsigned long long)hi << 32) | lo);
}

__device__ __forceinline__ unsigned wait_flag(const unsigned* flag, unsigned want, unsigned* tmo, unsigned call) {
  unsigned v = __builtin_amdgcn_readfirstlane(__hip_atomic_load(flag, __ATOMIC_RELAXED, __HIP_MEMORY_SCOPE_SYSTEM));
  if (v >= want) return v;
  const unsigned long long t0 = __builtin_amdgcn_s_memrealtime();
  for (;;) {
    __builtin_amdgcn_s_sleep(1);
    v = __builtin_amdgcn_readfirstlane(__hip_atomic_load(flag, __ATOMIC_RELAXED, __HIP_MEMORY_SCOPE_SYSTEM));
    if (v >= want) return v;
    if (__builtin_amdgcn_readfirstlane(__hip_atomic_load(tmo, __ATOMIC_RELAXED, __HIP_MEMORY_SCOPE_AGENT)) == call ||
        __builtin_amdgcn_s_memrealtime() - t0 > DEADLINE_TICKS) {
      __hip_atomic_store(tmo, call, __ATOMIC_RELAXED, __HIP_MEMORY_SCOPE_AGENT);
      return 0xffffffffu;
    }
  }
}

__global__ void __launch_bounds__(512, 2) gemm_ag_kernel(Params p) {
  __shared__ __attribute__((aligned(16))) unsigned char lds[LDS_BYTES];
  LAS unsigned char* L = (LAS unsigned char*)lds;
  const unsigned lbase = (unsigned)(size_t)L;

  const int tid = threadIdx.x;
  const int wave = __builtin_amdgcn_readfirstlane(tid >> 6);
  const int lane = tid & 63;
  const int b = blockIdx.x;
  const int xcd = b & 7, idx = b >> 3;
  const int srel = idx >> 2;
  const int cb = (xcd & 1) * 4 + (idx & 3);
  const int rb = xcd >> 1;
  const int rank = p.rank & (NR - 1);
  const int src = (rank + srel) & (NR - 1);
  const bool is_local = (srel == 0);
  const bool is_pusher = is_local && (cb < NR - 1);
  const int peer = (rank + 1 + cb) & (NR - 1);

  unsigned char* ws = p.ws[rank];
  unsigned* call_ctr = (unsigned*)(ws + WS_CALL + (size_t)b * 128);
  unsigned* tmo = (unsigned*)(ws + WS_TMO);
  const unsigned call =
      __builtin_amdgcn_readfirstlane(__hip_atomic_load(call_ctr, __ATOMIC_RELAXED, __HIP_MEMORY_SCOPE_AGENT)) + 1u;
  const float scale = __builtin_bit_cast(float, __builtin_amdgcn_readfirstlane(__builtin_bit_cast(int, p.sx[0] * p.sw[0])));
  const unsigned seq0 = call * 64u;
  const size_t slot_off = WS_STAGE + (size_t)(call & 1u) * IMG_SLOT;

  const unsigned char* abase;
  unsigned a_voff[2];
  unsigned a_step;
  if (is_local) {
    abase = p.x + (size_t)rb * BM * KT;
    a_step = BK;
#pragma unroll
    for (int j = 0; j < 2; ++j) {
      const int m = wave * 16 + j * 8 + (lane >> 3);
      const int ch = (lane & 7) ^ ((m >> 1) & 7);
      a_voff[j] = (unsigned)(m * KT + ch * 16);
    }
  } else {
    abase = ws + slot_off + (size_t)src * IMG_SRC + (size_t)rb * IMG_RB;
    a_step = A_BYTES;
#pragma unroll
    for (int j = 0; j < 2; ++j) a_voff[j] = (unsigned)(wave * 2048 + j * 1024 + lane * 16);
  }
  const auto rsA = __builtin_amdgcn_make_buffer_rsrc((void*)uni_ptr(abase), 0, (int)(BM * KT), 0x00020000);
  const unsigned char* bbase = p.w + (size_t)rank * NS + (size_t)cb * BN;
  const auto rsB = __builtin_amdgcn_make_buffer_rsrc((void*)uni_ptr(bbase), 0, (int)((size_t)KT * NT - (size_t)rank * NS - (size_t)cb * BN), 0x00020000);
  unsigned b_voff[2];
#pragma unroll
  for (int j = 0; j < 2; ++j) {
    const int k = wave * 16 + j * 4 + (lane >> 4);
    const int f = (k & 7) | (((k >> 4) & 1) << 3);
    const int ch = (lane & 15) ^ f;
    b_voff[j] = (unsigned)(k * NT + ch * 16);
  }
  unsigned char* pws = p.ws[is_pusher ? peer : rank];
  const auto rsP = __builtin_amdgcn_make_buffer_rsrc(
      (void*)uni_ptr(pws + slot_off + (size_t)rank * IMG_SRC + (size_t)rb * IMG_RB), 0, (int)IMG_RB, 0x00020000);
  unsigned* pflag = (unsigned*)(pws + WS_FLAG + (size_t)(rank * NRB + rb) * 128);
  const unsigned* myflag = (const unsigned*)(ws + WS_FLAG + (size_t)(src * NRB + rb) * 128);

  auto stage_load = [&](int s) {
    const int st = s % NSTAGE;
    LAS unsigned char* la = L + st * STAGE_BYTES + wave * 2048;
    LAS unsigned char* lb = L + st * STAGE_BYTES + A_BYTES + wave * 4096;
    const unsigned ao = (unsigned)s * a_step;
    const unsigned bo = (unsigned)s * (unsigned)(BK * NT);
    __builtin_amdgcn_raw_ptr_buffer_load_lds(rsA, (LAS void*)la, 16, a_voff[0], ao, 0, AUX_SYS);
    __builtin_amdgcn_raw_ptr_buffer_load_lds(rsA, (LAS void*)(la + 1024), 16, a_voff[1], ao, 0, AUX_SYS);
    __builtin_amdgcn_raw_ptr_buffer_load_lds(rsB, (LAS void*)lb, 16, b_voff[0], bo, 0, 0);
    __builtin_amdgcn_raw_ptr_buffer_load_lds(rsB, (LAS void*)(lb + 1024), 16, b_voff[1], bo, 0, 0);
    __builtin_amdgcn_raw_ptr_buffer_load_lds(rsB, (LAS void*)(lb + 2048), 16, b_voff[0], bo + 8u * NT, 0, 0);
    __builtin_amdgcn_raw_ptr_buffer_load_lds(rsB, (LAS void*)(lb + 3072), 16, b_voff[1], bo + 8u * NT, 0, 0);
  };

  const int wm = wave >> 2, wn = wave & 3;
  const int fr = lane & 15, g = lane >> 4;
  const unsigned xa0 = (unsigned)((wm * 64 + fr) * BK + ((g ^ ((fr >> 1) & 7)) * 16));
  unsigned wb[4];
  {
    const int q = fr >> 1, ph = fr & 1;
    const int f = q | ((g & 1) << 3);
#pragma unroll
    for (int ni = 0; ni < 4; ++ni)
      wb[ni] = (unsigned)(A_BYTES + (16 * g + q) * BN + (((wn * 4 + ni) ^ f) * 16) + ph * 8);
  }

  v4f acc[4][4];
#pragma unroll
  for (int i = 0; i < 4; ++i)
#pragma unroll
    for (int j = 0; j < 4; ++j) acc[i][j] = (v4f){0.f, 0.f, 0.f, 0.f};

  const bool poller = (!is_local) && (wave == 0);
  unsigned known = 0;

  if (poller) known = wait_flag(myflag, seq0 + 1u, tmo, call);
  if (!is_local) BAR();
  stage_load(0);
  if (poller && known < seq0 + 2u) known = wait_flag(myflag, seq0 + 2u, tmo, call);
  if (!is_local) BAR();
  stage_load(1);

  for (int i = 0; i < NSTEP; ++i) {
    if (is_pusher) {
      if (i < NSTEP - 1) WAIT_V(10); else WAIT_V(4);
    } else {
      if (i < NSTEP - 1) WAIT_V(6); else WAIT_V(0);
    }
    if (poller && i + 2 < NSTEP) {
      if (known < seq0 + (unsigned)(i + 3)) known = wait_flag(myflag, seq0 + (unsigned)(i + 3), tmo, call);
      if (i + 3 == NSTEP) {
        const unsigned t = __builtin_amdgcn_readfirstlane(__hip_atomic_load(tmo, __ATOMIC_RELAXED, __HIP_MEMORY_SCOPE_AGENT));
        if (lane == 0) *(LAS unsigned*)(L + NSTAGE * STAGE_BYTES) = (t == call) ? 1u : 0u;
        asm volatile("s_waitcnt lgkmcnt(0)" ::: "memory");
      }
    }
    BAR();
    if (is_pusher && wave == 0 && i >= 3 && lane == 0)
      __hip_atomic_store(pflag, seq0 + (unsigned)(i - 2), __ATOMIC_RELAXED, __HIP_MEMORY_SCOPE_SYSTEM);
    if (i + 2 < NSTEP) stage_load(i + 2);
    const unsigned so = (unsigned)((i % NSTAGE) * STAGE_BYTES);
    if (is_pusher) {
      const u32x4 d0 = *(const LAS u32x4*)(L + so + wave * 2048 + lane * 16);
      const u32x4 d1 = *(const LAS u32x4*)(L + so + wave * 2048 + 1024 + lane * 16);
      __builtin_amdgcn_raw_buffer_store_b128(d0, rsP, (unsigned)(wave * 2048 + lane * 16), (unsigned)i * A_BYTES, AUX_SYS);
      __builtin_amdgcn_raw_buffer_store_b128(d1, rsP, (unsigned)(wave * 2048 + 1024 + lane * 16), (unsigned)i * A_BYTES, AUX_SYS);
    }
    const unsigned axa = lbase + so + xa0, axb = lbase + so + (xa0 ^ 64u);
    const unsigned aw0 = lbase + so + wb[0], aw1 = lbase + so + wb[1], aw2 = lbase + so + wb[2], aw3 = lbase + so + wb[3];
    v2i w0a, w0b, w0c, w0d, w1a, w1b, w1c, w1d, w2a, w2b, w2c, w2d, w3a, w3b, w3c, w3d;
    v4i x0l, x0h, x1l, x1h, x2l, x2h, x3l, x3h;
#define TR8(dst, addr, off) asm volatile("ds_read_b64_tr_b8 %0, %1 offset:%2" : "=v"(dst) : "v"(addr), "i"(off))
#define RD128(dst, addr, off) asm volatile("ds_read_b128 %0, %1 offset:%2" : "=v"(dst) : "v"(addr), "i"(off))
#define TRW(a, b, c, d, addr) do { TR8(a, addr, 0); TR8(b, addr, 8 * BN); TR8(c, addr, 64 * BN); TR8(d, addr, 72 * BN); } while (0)
#define MF(ni, mi, W, X) acc[ni][mi] = __builtin_amdgcn_mfma_scale_f32_16x16x128_f8f6f4(W, X, acc[ni][mi], 1  , 0  , 0, 0, 0, 0)
#define CAT4(a, b, c, d) ((v8i){a[0], a[1], b[0], b[1], c[0], c[1], d[0], d[1]})
#define CAT2(l, h) ((v8i){l[0], l[1], l[2], l[3], h[0], h[1], h[2], h[3]})
    TRW(w0a, w0b, w0c, w0d, aw0);
    RD128(x0l, axa, 0); RD128(x0h, axb, 0);
    RD128(x1l, axa, 2048); RD128(x1h, axb, 2048);
    RD128(x2l, axa, 4096); RD128(x2h, axb, 4096);
    RD128(x3l, axa, 6144); RD128(x3h, axb, 6144);
    TRW(w1a, w1b, w1c, w1d, aw1);
    TRW(w2a, w2b, w2c, w2d, aw2);
    TRW(w3a, w3b, w3c, w3d, aw3);
    asm volatile("s_waitcnt lgkmcnt(12)"
                 : "+v"(w0a), "+v"(w0b), "+v"(w0c), "+v"(w0d), "+v"(x0l), "+v"(x0h), "+v"(x1l), "+v"(x1h),
                   "+v"(x2l), "+v"(x2h), "+v"(x3l), "+v"(x3h));
    const v8i xf0 = CAT2(x0l, x0h), xf1 = CAT2(x1l, x1h), xf2 = CAT2(x2l, x2h), xf3 = CAT2(x3l, x3h);
    { const v8i wf = CAT4(w0a, w0b, w0c, w0d); MF(0, 0, wf, xf0); MF(0, 1, wf, xf1); MF(0, 2, wf, xf2); MF(0, 3, wf, xf3); }
    __builtin_amdgcn_sched_barrier(0);
    asm volatile("s_waitcnt lgkmcnt(8)" : "+v"(w1a), "+v"(w1b), "+v"(w1c), "+v"(w1d));
    { const v8i wf = CAT4(w1a, w1b, w1c, w1d); MF(1, 0, wf, xf0); MF(1, 1, wf, xf1); MF(1, 2, wf, xf2); MF(1, 3, wf, xf3); }
    __builtin_amdgcn_sched_barrier(0);
    asm volatile("s_waitcnt lgkmcnt(4)" : "+v"(w2a), "+v"(w2b), "+v"(w2c), "+v"(w2d));
    { const v8i wf = CAT4(w2a, w2b, w2c, w2d); MF(2, 0, wf, xf0); MF(2, 1, wf, xf1); MF(2, 2, wf, xf2); MF(2, 3, wf, xf3); }
    __builtin_amdgcn_sched_barrier(0);
    asm volatile("s_waitcnt lgkmcnt(0)" : "+v"(w3a), "+v"(w3b), "+v"(w3c), "+v"(w3d));
    { const v8i wf = CAT4(w3a, w3b, w3c, w3d); MF(3, 0, wf, xf0); MF(3, 1, wf, xf1); MF(3, 2, wf, xf2); MF(3, 3, wf, xf3); }
  }

  if (is_pusher) {
    WAIT_V(4); BAR();
    if (wave == 0 && lane == 0) __hip_atomic_store(pflag, seq0 + (unsigned)(NSTEP - 2), __ATOMIC_RELAXED, __HIP_MEMORY_SCOPE_SYSTEM);
    WAIT_V(2); BAR();
    if (wave == 0 && lane == 0) __hip_atomic_store(pflag, seq0 + (unsigned)(NSTEP - 1), __ATOMIC_RELAXED, __HIP_MEMORY_SCOPE_SYSTEM);
    WAIT_V(0); BAR();
    if (wave == 0 && lane == 0) __hip_atomic_store(pflag, seq0 + (unsigned)NSTEP, __ATOMIC_RELAXED, __HIP_MEMORY_SCOPE_SYSTEM);
  }

  bool bad = false;
  if (!is_local) bad = (*(const LAS unsigned*)(L + NSTAGE * STAGE_BYTES)) != 0u;
  float* orow = p.out + (size_t)(src * MS + rb * BM + wm * 64 + fr) * NS + cb * BN + wn * 64 + g * 4;
#pragma unroll
  for (int mi = 0; mi < 4; ++mi) {
#pragma unroll
    for (int ni = 0; ni < 4; ++ni) {
      v4f o;
#pragma unroll
      for (int r = 0; r < 4; ++r) {
        const float y = acc[ni][mi][r] * scale;
        const float c = __builtin_amdgcn_fmed3f(y, -60.0f, 60.0f);
        const float e = __builtin_amdgcn_exp2f(c * -1.4426950408889634f);
        o[r] = y * __builtin_amdgcn_rcpf(1.0f + e);
      }
      if (bad) o = (v4f){__builtin_nanf(""), __builtin_nanf(""), __builtin_nanf(""), __builtin_nanf("")};
      *(v4f*)(orow + (size_t)mi * 16 * NS + ni * 16) = o;
    }
  }
  if (tid == 0) __hip_atomic_store(call_ctr, call, __ATOMIC_RELAXED, __HIP_MEMORY_SCOPE_AGENT);
}

}

extern "C" void kernel_launch(int rank, int nranks, ncclComm_t comm,
                              hipStream_t stream,
                              void* const* d_in, const int* in_sizes, int n_in,
                              void* d_out, int out_size,
                              const int* in_dtypes, int out_dtype,
                              void* const* d_out_all,
                              void* const* d_ws_all, size_t ws_bytes) {
  (void)comm; (void)in_dtypes; (void)out_dtype; (void)d_out_all;
  if (nranks != NR || n_in < 4 || (unsigned)rank >= (unsigned)NR || ws_bytes < WS_TOTAL ||
      in_sizes[0] != MS * KT || in_sizes[1] != KT * NT || out_size != MT * NS) {
    fprintf(stderr, "unexpected problem shape\n");
    return;
  }
  Params p;
  p.x = (const unsigned char*)d_in[0];
  p.w = (const unsigned char*)d_in[1];
  p.sx = (const float*)d_in[2];
  p.sw = (const float*)d_in[3];
  p.out = (float*)d_out;
  for (int r = 0; r < NR; ++r) p.ws[r] = (unsigned char*)d_ws_all[r];
  p.rank = rank;
  hipLaunchKernelGGL(gemm_ag_kernel, dim3(NWG), dim3(512), 0, stream, p);
  HIP_CHECK(hipGetLastError());
}
```

```cpp
#include <hip/hip_runtime.h>
#include <rccl/rccl.h>
#include <cstdio>
#include <cstdint>

#define HIP_CHECK(expr) do { hipError_t _e = (expr); if (_e != hipSuccess) { \
    fprintf(stderr, "HIP error %s at %s:%d\n", hipGetErrorString(_e), __FILE__, __LINE__); \
    return; } } while (0)

namespace {

#define LAS __attribute__((address_space(3)))
typedef int v2i __attribute__((ext_vector_type(2)));
typedef int v4i __attribute__((ext_vector_type(4)));
typedef int v8i __attribute__((ext_vector_type(8)));
typedef unsigned u32x4 __attribute__((ext_vector_type(4)));
typedef float v4f __attribute__((ext_vector_type(4)));

constexpr int NR = 8;
constexpr int MT = 4096, KT = 4096, NT = 16384;
constexpr int MS = MT / NR;
constexpr int NS = NT / NR;
constexpr int BM = 128, BN = 256, BK = 128;
constexpr int NSTEP = KT / BK;
constexpr int NRB = MS / BM;
constexpr int NCB = NS / BN;
constexpr int NWG = (MT / BM) * NCB;
constexpr int A_BYTES = BM * BK;
constexpr int B_BYTES = BK * BN;
constexpr int S_BYTES = 2 * 16 * BK;
constexpr int S_OFF = A_BYTES + B_BYTES;
constexpr int STAGE_BYTES = A_BYTES + B_BYTES + S_BYTES;
constexpr int NSTAGE = 3;
constexpr int LDS_BYTES = NSTAGE * STAGE_BYTES + 128;

constexpr size_t WS_CALL = 0;
constexpr size_t WS_TMO = WS_CALL + (size_t)NWG * 128;
constexpr size_t WS_FLAG = WS_TMO + 128;
constexpr size_t WS_STAGE = 1u << 20;
constexpr size_t IMG_RB = (size_t)NSTEP * A_BYTES;
constexpr size_t IMG_SRC = IMG_RB * NRB;
constexpr size_t IMG_SLOT = IMG_SRC * NR;
constexpr size_t WS_TOTAL = WS_STAGE + 2 * IMG_SLOT;
static_assert(WS_FLAG + (size_t)NR * NRB * 4 * 128 <= WS_STAGE, "control block");

constexpr unsigned long long DEADLINE_TICKS = 40000000ull;
#define POLL_SLEEP 1
#define OUT_AUX 16
#define X_AUX 17
constexpr int AUX_SYS = 17;

struct Params {
  const unsigned char* x;
  const unsigned char* w;
  const float* sx;
  const float* sw;
  float* out;
  unsigned char* ws[NR];
  int rank;
};
#define STAMP(k) do { } while (0)
#define CSTAMP(k) do { } while (0)

#define WAIT_V(n) asm volatile("s_waitcnt vmcnt(" #n ")" ::: "memory")
#define BAR() do { asm volatile("" ::: "memory"); __builtin_amdgcn_s_barrier(); asm volatile("" ::: "memory"); } while (0)

__device__ __forceinline__ const void* uni_ptr(const void* p) {
  const unsigned long long v = (unsigned long long)p;
  const unsigned lo = __builtin_amdgcn_readfirstlane((unsigned)v);
  const unsigned hi = __builtin_amdgcn_readfirstlane((unsigned)(v >> 32));
  return (const void*)(((unsigned long long)hi << 32) | lo);
}

__device__ __forceinline__ void wait_vm(int n) { asm volatile("s_waitcnt vmcnt(%0)" :: "i"(n) : "memory"); }

__device__ __forceinline__ unsigned wait_flag(const unsigned* flag, unsigned want, unsigned* tmo, unsigned call) {
  unsigned v = __builtin_amdgcn_readfirstlane(__hip_atomic_load(flag, __ATOMIC_RELAXED, __HIP_MEMORY_SCOPE_SYSTEM));
  if (v >= want) return v;
  const unsigned long long t0 = __builtin_amdgcn_s_memrealtime();
  for (;;) {
    __builtin_amdgcn_s_sleep(POLL_SLEEP);
    v = __builtin_amdgcn_readfirstlane(__hip_atomic_load(flag, __ATOMIC_RELAXED, __HIP_MEMORY_SCOPE_SYSTEM));
    if (v >= want) return v;
    if (__builtin_amdgcn_readfirstlane(__hip_atomic_load(tmo, __ATOMIC_RELAXED, __HIP_MEMORY_SCOPE_AGENT)) == call ||
        __builtin_amdgcn_s_memrealtime() - t0 > DEADLINE_TICKS) {
      __hip_atomic_store(tmo, call, __ATOMIC_RELAXED, __HIP_MEMORY_SCOPE_AGENT);
      asm volatile("s_waitcnt vmcnt(0)" ::: "memory");
      return 0xffffffffu;
    }
  }
}

#define PUSH_D 1
#define PUSH_PP (PUSH_D + 3)

__global__ void __launch_bounds__(512, 2) gemm_ag_kernel(Params p) {
  __shared__ __attribute__((aligned(16))) unsigned char lds[LDS_BYTES];
  LAS unsigned char* L = (LAS unsigned char*)lds;
  const unsigned lbase = (unsigned)(size_t)L;

  const int tid = threadIdx.x;
  const int wave = __builtin_amdgcn_readfirstlane(tid >> 6);
  const int lane = tid & 63;
  const int b = blockIdx.x;
  const int xcd = b & 7, idx = b >> 3;
  const int rank = p.rank & (NR - 1);

  STAMP(0);
  unsigned char* ws = p.ws[rank];
  unsigned* call_ctr = (unsigned*)(ws + WS_CALL + (size_t)b * 128);
  unsigned* tmo = (unsigned*)(ws + WS_TMO);
  const unsigned call_raw = __hip_atomic_load(call_ctr, __ATOMIC_RELAXED, __HIP_MEMORY_SCOPE_AGENT);
#define CALL_STATE() \
  const unsigned call = __builtin_amdgcn_readfirstlane(call_raw) + 1u; \
  const unsigned seq0 = call * 64u;   \
  const size_t slot_off = WS_STAGE + (size_t)(call & 1u) * IMG_SLOT

  if (b < 4 * (NR - 1)) {
    if (wave < 4) {
      const int peer = (rank + 1 + b % (NR - 1)) & (NR - 1);
      const int prb = b / (NR - 1);
      const auto rsX = __builtin_amdgcn_make_buffer_rsrc((void*)uni_ptr(p.x + (size_t)(prb * BM + wave * 32) * KT), 0,
                                                         32 * KT, 0x00020000);
      unsigned voff[2];
#pragma unroll
      for (int jj = 0; jj < 2; ++jj) {
        const int m = jj * 8 + (lane >> 3);
        voff[jj] = (unsigned)(m * KT + (((lane & 7) ^ ((m >> 1) & 7)) * 16));
      }
      const unsigned vo = (unsigned)(wave * 4096 + lane * 16);
      constexpr int NP = NSTEP;
      constexpr int PP = PUSH_PP;
      u32x4 d[PP][4];
#define PLOAD(slot, n_) do { const unsigned o_ = (unsigned)(n_) * BK; \
        d[slot][0] = __builtin_amdgcn_raw_buffer_load_b128(rsX, voff[0], o_, 0); \
        d[slot][1] = __builtin_amdgcn_raw_buffer_load_b128(rsX, voff[1], o_, 0); \
        d[slot][2] = __builtin_amdgcn_raw_buffer_load_b128(rsX, voff[0], o_ + 16u * KT, 0); \
        d[slot][3] = __builtin_amdgcn_raw_buffer_load_b128(rsX, voff[1], o_ + 16u * KT, 0); } while (0)
#define PFLAG(n_) do { if (lane == 0) __hip_atomic_store(pflag, seq0 + (unsigned)((n_) + 1), \
        __ATOMIC_RELAXED, __HIP_MEMORY_SCOPE_SYSTEM); } while (0)
      PLOAD(0, 0);
      __builtin_amdgcn_sched_barrier(0);
      CALL_STATE();
      __builtin_amdgcn_sched_barrier(0);
#pragma unroll
      for (int n = 1; n < PP; ++n) PLOAD(n, n);
      unsigned char* pws = p.ws[peer];
      const auto rsP = __builtin_amdgcn_make_buffer_rsrc(
          (void*)uni_ptr(pws + slot_off + (size_t)rank * IMG_SRC + (size_t)prb * IMG_RB), 0, (int)IMG_RB, 0x00020000);
      unsigned* pflag = (unsigned*)(pws + WS_FLAG + (size_t)((rank * NRB + prb) * 4 + wave) * 128);
      STAMP(1);
#pragma unroll
      for (int n = 0; n < NP; ++n) {
        const unsigned po = (unsigned)n * A_BYTES;
        __builtin_amdgcn_raw_buffer_store_b128(d[n % PP][0], rsP, vo, po, AUX_SYS);
        __builtin_amdgcn_raw_buffer_store_b128(d[n % PP][1], rsP, vo + 1024, po, AUX_SYS);
        __builtin_amdgcn_raw_buffer_store_b128(d[n % PP][2], rsP, vo + 2048, po, AUX_SYS);
        __builtin_amdgcn_raw_buffer_store_b128(d[n % PP][3], rsP, vo + 3072, po, AUX_SYS);
        if (n >= PUSH_D) {
          int c = 4 * PUSH_D;
#pragma unroll
          for (int q = n - PUSH_D; q < n; ++q) c += (q >= PUSH_D ? 1 : 0) + (q + PP < NP ? 4 : 0);
          wait_vm(c);
          PFLAG(n - PUSH_D);
        }
        if (n + PP < NP) PLOAD(n % PP, n + PP);
      }
#pragma unroll
      for (int r = PUSH_D - 1; r >= 0; --r) {
        wait_vm(4 * r + PUSH_D);
        PFLAG(NP - 1 - r);
      }
    }
    STAMP(34);
    STAMP(35);
    if (tid == 0) __hip_atomic_store(call_ctr, call_raw + 1u, __ATOMIC_RELAXED, __HIP_MEMORY_SCOPE_AGENT);
    STAMP(36);
    return;
  }

  if (b < 4 * NR) {
    if (tid == 0) __hip_atomic_store(call_ctr, call_raw + 1u, __ATOMIC_RELAXED, __HIP_MEMORY_SCOPE_AGENT);
    return;
  }

  const int srel = 1 + ((idx - 4) >> 2);
  const int cb = (xcd & 1) * 4 + ((idx - 4) & 3);
  const int rb = xcd >> 1;
  const int src = (rank + srel) & (NR - 1);
  const int wm = wave >> 2, wn = wave & 3;
  const int sq = rb * (NR - 1) + (srel - 1);
  const int srowA = sq * 16, srowB = sq < 4 ? 448 + sq * 16 : srowA;
  const bool strip = wm == 0 || sq < 4;
  const float scale = __builtin_bit_cast(float, __builtin_amdgcn_readfirstlane(__builtin_bit_cast(int, p.sx[0] * p.sw[0])));

  unsigned xvoff[2], wvoff[2], svoff;
#pragma unroll
  for (int j = 0; j < 2; ++j) {
    xvoff[j] = (unsigned)(wave * 2048 + j * 1024 + lane * 16);
    const int k = wave * 16 + j * 4 + (lane >> 4);
    wvoff[j] = (unsigned)(k * NT + (((lane & 15) ^ ((k & 7) | (((k >> 4) & 1) << 3))) * 16));
  }
  {
    const int m = (wave & 3) * 4 + ((lane >> 3) & 3);
    svoff = (unsigned)(((wm ? srowB : srowA) + m) * KT + (((lane & 7) ^ ((m >> 1) & 7)) * 16));
  }
  CALL_STATE();
  const auto rsX = __builtin_amdgcn_make_buffer_rsrc(
      (void*)uni_ptr(ws + slot_off + (size_t)src * IMG_SRC + (size_t)rb * IMG_RB), 0, (int)IMG_RB, 0x00020000);
  const auto rsS = __builtin_amdgcn_make_buffer_rsrc((void*)uni_ptr(p.x), 0, MS * KT, 0x00020000);
  const auto rsW = __builtin_amdgcn_make_buffer_rsrc((void*)uni_ptr(p.w + (size_t)rank * NS + (size_t)cb * BN), 0,
                                                     (int)((size_t)KT * NT - (size_t)rank * NS - (size_t)cb * BN), 0x00020000);
  const unsigned* myflag = (const unsigned*)(ws + WS_FLAG + (size_t)((src * NRB + rb) * 4 + (wave >> 1)) * 128);

#define CLAMP(s) ((s) < NSTEP ? (s) : NSTEP - 1)
#define DMAX(so, s, k) __builtin_amdgcn_raw_ptr_buffer_load_lds(rsX, (LAS void*)(L + (so) + wave * 2048 + (k) * 1024), 16, \
    xvoff[k], (unsigned)CLAMP(s) * A_BYTES, 0, X_AUX)
#define DMAW(so, s, k) __builtin_amdgcn_raw_ptr_buffer_load_lds(rsW, (LAS void*)(L + (so) + A_BYTES + wave * 4096 + (k) * 1024), 16, \
    wvoff[(k) & 1], (unsigned)CLAMP(s) * (unsigned)(BK * NT) + (unsigned)((k) >> 1) * (8u * NT), 0, 0)
#define DMAS(so, s) do { if (lane < 32) __builtin_amdgcn_raw_ptr_buffer_load_lds(rsS, (LAS void*)(L + (so) + S_OFF + wave * 512), 16, \
    svoff, (unsigned)CLAMP(s) * BK, 0, 0); } while (0)
  auto load_x = [&](int s) { const unsigned so = (unsigned)((s % NSTAGE) * STAGE_BYTES); DMAX(so, s, 0); DMAX(so, s, 1); };
  auto load_w = [&](int s) { const unsigned so = (unsigned)((s % NSTAGE) * STAGE_BYTES);
    DMAW(so, s, 0); DMAW(so, s, 1); DMAW(so, s, 2); DMAW(so, s, 3); DMAS(so, s); };

  const int fr = lane & 15, g = lane >> 4;
  const unsigned xa0 = lbase + (unsigned)((wm * 64 + fr) * BK + ((g ^ ((fr >> 1) & 7)) * 16));
  const unsigned sa0 = lbase + (unsigned)(S_OFF + (wm * 16 + fr) * BK + ((g ^ ((fr >> 1) & 7)) * 16));
  unsigned wb[4];
  {
    const int q = fr >> 1, ph = fr & 1;
    const int f = q | ((g & 1) << 3);
#pragma unroll
    for (int ni = 0; ni < 4; ++ni)
      wb[ni] = lbase + (unsigned)(A_BYTES + (16 * g + q) * BN + (((wn * 4 + ni) ^ f) * 16) + ph * 8);
  }

  constexpr int ALL = NSTEP + 8;
  int known = 0;
  int xi = 0;
  int pend = 0;
#define UNI(v) v = __builtin_amdgcn_readfirstlane(v)
  LAS unsigned* actl = (LAS unsigned*)(L + NSTAGE * STAGE_BYTES + 32 + wave * 8);
  const unsigned actl_a = lbase + NSTAGE * STAGE_BYTES + 32 + wave * 8;

  auto pre_sync = [&](int j) {
    UNI(xi); UNI(pend);
    if (__builtin_expect(pend == 0 && xi == j + 2, 1)) { WAIT_V(7); return; }
    UNI(known);
    if (pend) {
      WAIT_V(4);
      unsigned pv;
      asm volatile("ds_read_b32 %0, %1\n\ts_waitcnt lgkmcnt(0)" : "=v"(pv) : "v"(actl_a) : "memory");
      const int k = (int)(__builtin_amdgcn_readfirstlane(pv) - seq0);
      known = k > known ? (k < NSTEP ? k : ALL) : known;
      pend = 0;
    } else if (xi > j) {
      WAIT_V(5);
    }
    if (xi <= j) {
      if (known <= j) {
        const unsigned v = wait_flag(myflag, seq0 + (unsigned)(j + 1), tmo, call);
        const int k = (int)(v - seq0);
        known = (v != 0xffffffffu && k < NSTEP) ? k : ALL;
      }
      load_x(j);
      xi = j + 1;
      if (known > j + 1) { load_x(j + 1); xi = j + 2; WAIT_V(2); } else WAIT_V(0);
    } else if (xi == j + 1 && known > j + 1) {
      load_x(j + 1);
      xi = j + 2;
    }
  };
  auto post_x = [&](int j) -> int {
    UNI(known); UNI(xi);
    if (__builtin_expect(xi == j + 2 && known > j + 2, 1)) { xi = j + 3; return 1; }
    return 0;
  };
  auto post_poll = [&](int j) {
    UNI(known);
    if (__builtin_expect(known < j + 6 || j == NSTEP - 1, 0)) {
      const int np = j < NSTEP - 1 ? 1 : 0;
      if (np && lane == 0) __builtin_amdgcn_global_load_lds(myflag, actl, 4, 0, AUX_SYS);
      if (!np && lane == 0) __builtin_amdgcn_global_load_lds(tmo, actl + 1, 4, 0, 16  );
      pend = np;
    }
  };

#define TR8(dst, addr, off) asm volatile("ds_read_b64_tr_b8 %0, %1 offset:%2" : "=v"(dst) : "v"(addr), "i"(off))
#define RD128(dst, addr, off) asm volatile("ds_read_b128 %0, %1 offset:%2" : "=v"(dst) : "v"(addr), "i"(off))
#define TRW(W, addr) do { const unsigned a_ = (addr); TR8(W##a, a_, 0); TR8(W##b, a_, 8 * BN); TR8(W##c, a_, 64 * BN); TR8(W##d, a_, 72 * BN); } while (0)
#define RDX(X, so_) do { const unsigned a_ = xa0 + (so_), b_ = (xa0 ^ 64u) + (so_); \
    RD128(X##0l, a_, 0); RD128(X##0h, b_, 0); RD128(X##1l, a_, 2048); RD128(X##1h, b_, 2048); \
    RD128(X##2l, a_, 4096); RD128(X##2h, b_, 4096); RD128(X##3l, a_, 6144); RD128(X##3h, b_, 6144); \
    RD128(X##4l, sa0 + (so_), 0); RD128(X##4h, (sa0 ^ 64u) + (so_), 0); } while (0)
#define CAT4(W) ((v8i){W##a[0], W##a[1], W##b[0], W##b[1], W##c[0], W##c[1], W##d[0], W##d[1]})
#define CAT2(l, h) ((v8i){l[0], l[1], l[2], l[3], h[0], h[1], h[2], h[3]})
#define MF1(ni, mi, Wv, Xv) asm volatile("v_mfma_f32_16x16x128_f8f6f4 %0, %1, %2, %0 cbsz:1" : "+a"(acc[ni][mi]) : "v"(Wv), "v"(Xv))
#define PINW1(n, W) asm volatile("s_waitcnt lgkmcnt(" #n ")" : "+v"(W##a), "+v"(W##b), "+v"(W##c), "+v"(W##d))
#define PINW2(W, V) asm volatile("s_waitcnt lgkmcnt(0)" : "+v"(W##a), "+v"(W##b), "+v"(W##c), "+v"(W##d), "+v"(V##a), "+v"(V##b), \
    "+v"(V##c), "+v"(V##d) :: "memory")
#define PINX(X, W) asm volatile("s_waitcnt lgkmcnt(0)" : "+v"(W##a), "+v"(W##b), "+v"(W##c), "+v"(W##d), "+v"(X##0l), "+v"(X##0h), \
    "+v"(X##1l), "+v"(X##1h), "+v"(X##2l), "+v"(X##2h), "+v"(X##3l), "+v"(X##3h), "+v"(X##4l), "+v"(X##4h))
#define SB0() __builtin_amdgcn_sched_barrier(0)
#define MFX(ni, mi, W, X) do { SB0(); MF1(ni, mi, CAT4(W), CAT2(X##mi##l, X##mi##h)); SB0(); } while (0)
#define MFS(ni, W, X) do { if (strip) MFX(ni, 4, W, X); } while (0)
#define KHALF1(XC, so_) \
    const unsigned a1_ = wb[1] + (so_), a2_ = wb[2] + (so_), a3_ = wb[3] + (so_); \
    MFX(0, 0, w0, XC); TR8(w1a, a1_, 0); TR8(w1b, a1_, 8 * BN); TR8(w1c, a1_, 64 * BN); \
    MFX(0, 1, w0, XC); TR8(w1d, a1_, 72 * BN); TR8(w2a, a2_, 0); TR8(w2b, a2_, 8 * BN); \
    MFX(0, 2, w0, XC); TR8(w2c, a2_, 64 * BN); TR8(w2d, a2_, 72 * BN); TR8(w3a, a3_, 0); \
    MFX(0, 3, w0, XC); TR8(w3b, a3_, 8 * BN); TR8(w3c, a3_, 64 * BN); TR8(w3d, a3_, 72 * BN); \
    MFS(0, w0, XC); \
    PINW1(8, w1);
#define KSTEP(i, XC, XN, so_, sn_, s2_) do { \
    KHALF1(XC, so_) \
    MFX(1, 0, w1, XC); DMAW(s2_, (i) + 2, 1); \
    MFX(1, 1, w1, XC); DMAW(s2_, (i) + 2, 2); \
    MFX(1, 2, w1, XC); DMAW(s2_, (i) + 2, 3); \
    MFX(1, 3, w1, XC); DMAS(s2_, (i) + 2); \
    MFS(1, w1, XC); \
    pre_sync((i) + 1); \
    PINW2(w2, w3); \
    BAR(); STAMP(3 + (i)); \
    const int nx_ = post_x((i) + 1); \
    const unsigned xl_ = xa0 + (sn_), xh_ = (xa0 ^ 64u) + (sn_), a0_ = wb[0] + (sn_); \
    MFX(2, 0, w2, XC); RD128(XN##0l, xl_, 0); RD128(XN##0h, xh_, 0); TR8(w0a, a0_, 0); \
    MFX(2, 1, w2, XC); TR8(w0b, a0_, 8 * BN); TR8(w0c, a0_, 64 * BN); TR8(w0d, a0_, 72 * BN); \
    MFX(2, 2, w2, XC); RD128(XN##1l, xl_, 2048); RD128(XN##1h, xh_, 2048); \
    MFX(2, 3, w2, XC); RD128(XN##2l, xl_, 4096); RD128(XN##2h, xh_, 4096); \
    MFS(2, w2, XC); \
    MFX(3, 0, w3, XC); RD128(XN##3l, xl_, 6144); RD128(XN##3h, xh_, 6144); \
    MFX(3, 1, w3, XC); RD128(XN##4l, sa0 + (sn_), 0); RD128(XN##4h, (sa0 ^ 64u) + (sn_), 0); \
    if (nx_) { DMAX(so_, (i) + 3, 0); DMAX(so_, (i) + 3, 1); } \
    MFX(3, 2, w3, XC); DMAW(so_, (i) + 3, 0); \
    MFX(3, 3, w3, XC); post_poll((i) + 1); \
    MFS(3, w3, XC); \
    PINX(XN, w0); SB0(); \
  } while (0)
#define KLAST(XC, so_) do { \
    KHALF1(XC, so_) \
    MFX(1, 0, w1, XC); MFX(1, 1, w1, XC); MFX(1, 2, w1, XC); MFX(1, 3, w1, XC); MFS(1, w1, XC); \
    PINW2(w2, w3); \
    MFX(2, 0, w2, XC); MFX(2, 1, w2, XC); MFX(2, 2, w2, XC); MFX(2, 3, w2, XC); MFS(2, w2, XC); \
    MFX(3, 0, w3, XC); MFX(3, 1, w3, XC); MFX(3, 2, w3, XC); MFX(3, 3, w3, XC); MFS(3, w3, XC); \
  } while (0)

  v4f acc[4][5];
  v2i w0a, w0b, w0c, w0d, w1a, w1b, w1c, w1d, w2a, w2b, w2c, w2d, w3a, w3b, w3c, w3d;
  v4i xa0l, xa0h, xa1l, xa1h, xa2l, xa2h, xa3l, xa3h, xa4l, xa4h, xb0l, xb0h, xb1l, xb1h, xb2l, xb2h, xb3l, xb3h, xb4l, xb4h;

  load_w(0);
  load_w(1);
  STAMP(1);
#pragma unroll
  for (int i = 0; i < 4; ++i)
#pragma unroll
    for (int j = 0; j < 5; ++j) acc[i][j] = (v4f){0.f, 0.f, 0.f, 0.f};
  pre_sync(0);
  BAR();
  STAMP(2);
  CSTAMP(40);
  if (post_x(0)) load_x(2);
  DMAW(2 * STAGE_BYTES, 2, 0);
  post_poll(0);
  RDX(xa, 0u);
  TRW(w0, wb[0]);
  PINX(xa, w0);
  SB0();
  {
    unsigned s0 = 0, s1 = STAGE_BYTES, s2 = 2 * STAGE_BYTES;
    for (int i = 0; i < NSTEP - 2; i += 2) {
      KSTEP(i, xa, xb, s0, s1, s2);
      KSTEP(i + 1, xb, xa, s1, s2, s0);
      const unsigned t_ = s0; s0 = s2; s2 = s1; s1 = t_;
    }
    KSTEP(NSTEP - 2, xa, xb, s0, s1, s2);
    KLAST(xb, s1);
  }
  asm volatile("s_nop 15\n\ts_nop 15\n\ts_waitcnt vmcnt(0)"
               : "+a"(acc[0][0]), "+a"(acc[0][1]), "+a"(acc[0][2]), "+a"(acc[0][3]), "+a"(acc[0][4]),
                 "+a"(acc[1][0]), "+a"(acc[1][1]), "+a"(acc[1][2]), "+a"(acc[1][3]), "+a"(acc[1][4]) :: "memory");
  asm volatile(""
               : "+a"(acc[2][0]), "+a"(acc[2][1]), "+a"(acc[2][2]), "+a"(acc[2][3]), "+a"(acc[2][4]),
                 "+a"(acc[3][0]), "+a"(acc[3][1]), "+a"(acc[3][2]), "+a"(acc[3][3]), "+a"(acc[3][4]) :: "memory");
  STAMP(34);
  CSTAMP(41);
  unsigned tv;
  asm volatile("ds_read_b32 %0, %1 offset:4\n\ts_waitcnt lgkmcnt(0)" : "=v"(tv) : "v"(actl_a) : "memory");
  const float oscale = __builtin_amdgcn_readfirstlane(tv) == call ? __builtin_nanf("") : scale;
  STAMP(35);

  {
    constexpr float LOG2E = 1.4426950408889634f, LIM2 = 60.0f * LOG2E;
    const float nscale2 = -scale * LOG2E;
    const auto rsO = __builtin_amdgcn_make_buffer_rsrc((void*)uni_ptr(p.out), 0, MT * NS * 4, 0x00020000);
    const unsigned ocol = (unsigned)(cb * BN + wn * 64 + g * 4) * 4u;
    const unsigned ovo = ocol + (unsigned)(src * MS + rb * BM + wm * 64 + fr) * (unsigned)(NS * 4);
    const unsigned svo = ocol + (unsigned)(rank * MS + (wm ? srowB : srowA) + fr) * (unsigned)(NS * 4);
#pragma unroll
    for (int mi = 0; mi < 5; ++mi) {
      if (mi == 4 && !strip) break;
#pragma unroll
      for (int ni = 0; ni < 4; ++ni) {
        const v4f y = acc[ni][mi] * oscale;
        const v4f z = acc[ni][mi] * nscale2;
        v4f d;
#pragma unroll
        for (int r = 0; r < 4; ++r) d[r] = __builtin_amdgcn_exp2f(__builtin_amdgcn_fmed3f(z[r], -LIM2, LIM2));
        d = d + 1.0f;
#pragma unroll
        for (int r = 0; r < 4; ++r) d[r] = __builtin_amdgcn_rcpf(d[r]);
        const v4f o = y * d;
        __builtin_amdgcn_raw_buffer_store_b128(__builtin_bit_cast(u32x4, o), rsO,
                                               (mi < 4 ? ovo + (unsigned)(mi * 16 * NS * 4) : svo) + (unsigned)(ni * 64), 0, OUT_AUX);
      }
    }
  }
  if (tid == 0) __hip_atomic_store(call_ctr, call, __ATOMIC_RELAXED, __HIP_MEMORY_SCOPE_AGENT);
}

}

extern "C" void kernel_launch(int rank, int nranks, ncclComm_t comm,
                              hipStream_t stream,
                              void* const* d_in, const int* in_sizes, int n_in,
                              void* d_out, int out_size,
                              const int* in_dtypes, int out_dtype,
                              void* const* d_out_all,
                              void* const* d_ws_all, size_t ws_bytes) {
  (void)comm; (void)in_dtypes; (void)out_dtype; (void)d_out_all;
  if (nranks != NR || n_in < 4 || (unsigned)rank >= (unsigned)NR || ws_bytes < WS_TOTAL ||
      in_sizes[0] != MS * KT || in_sizes[1] != KT * NT || out_size != MT * NS) {
    fprintf(stderr, "unexpected problem shape\n");
    return;
  }
  Params p;
  p.x = (const unsigned char*)d_in[0];
  p.w = (const unsigned char*)d_in[1];
  p.sx = (const float*)d_in[2];
  p.sw = (const float*)d_in[3];
  p.out = (float*)d_out;
  for (int r = 0; r < NR; ++r) p.ws[r] = (unsigned char*)d_ws_all[r];
  p.rank = rank;
  hipLaunchKernelGGL(gemm_ag_kernel, dim3(NWG), dim3(512), 0, stream, p);
  HIP_CHECK(hipGetLastError());
}
```

```cpp
#include <hip/hip_runtime.h>
#include <rccl/rccl.h>
#include <cstdio>
#include <cstdint>

#define HIP_CHECK(expr) do { hipError_t _e = (expr); if (_e != hipSuccess) { \
    fprintf(stderr, "HIP error %s at %s:%d\n", hipGetErrorString(_e), __FILE__, __LINE__); \
    return; } } while (0)

namespace {

#define LAS __attribute__((address_space(3)))
typedef int v2i __attribute__((ext_vector_type(2)));
typedef int v4i __attribute__((ext_vector_type(4)));
typedef int v8i __attribute__((ext_vector_type(8)));
typedef unsigned u32x4 __attribute__((ext_vector_type(4)));
typedef float v4f __attribute__((ext_vector_type(4)));

constexpr int NR = 8;
constexpr int MT = 4096, KT = 4096, NT = 16384;
constexpr int MS = MT / NR;
constexpr int NS = NT / NR;
constexpr int BM = 128, BN = 256, BK = 128;
constexpr int NSTEP = KT / BK;
constexpr int NRB = MS / BM;
constexpr int NCB = NS / BN;
constexpr int NWG = (MT / BM) * NCB;
constexpr int A_BYTES = BM * BK;
constexpr int B_BYTES = BK * BN;
constexpr int S_BYTES = 2 * 16 * BK;
constexpr int S_OFF = A_BYTES + B_BYTES;
constexpr int STAGE_BYTES = A_BYTES + B_BYTES + S_BYTES;
constexpr int NSTAGE = 3;
constexpr int LDS_BYTES = NSTAGE * STAGE_BYTES + 128;

constexpr size_t WS_CALL = 0;
constexpr size_t WS_TMO = WS_CALL + (size_t)NWG * 128;
constexpr size_t WS_FLAG = WS_TMO + 128;
constexpr size_t WS_STAGE = 1u << 20;
constexpr size_t IMG_RB = (size_t)NSTEP * A_BYTES;
constexpr size_t IMG_SRC = IMG_RB * NRB;
constexpr size_t IMG_SLOT = IMG_SRC * NR;
constexpr size_t WS_TOTAL = WS_STAGE + 2 * IMG_SLOT;
static_assert(WS_FLAG + (size_t)NR * NRB * 4 * 128 <= WS_STAGE, "control block");

constexpr unsigned long long DEADLINE_TICKS = 40000000ull;
#define POLL_SLEEP 1
#define OUT_AUX 16
#define X_AUX 17
constexpr int AUX_SYS = 17;

struct Params {
  const unsigned char* x;
  const unsigned char* w;
  const float* sx;
  const float* sw;
  float* out;
  unsigned char* ws[NR];
  int rank;
};
#define STAMP(k) do { } while (0)
#define CSTAMP(k) do { } while (0)

#define WAIT_V(n) asm volatile("s_waitcnt vmcnt(" #n ")" ::: "memory")
#define BAR() do { asm volatile("" ::: "memory"); __builtin_amdgcn_s_barrier(); asm volatile("" ::: "memory"); } while (0)

__device__ __forceinline__ const void* uni_ptr(const void* p) {
  const unsigned long long v = (unsigned long long)p;
  const unsigned lo = __builtin_amdgcn_readfirstlane((unsigned)v);
  const unsigned hi = __builtin_amdgcn_readfirstlane((unsigned)(v >> 32));
  return (const void*)(((unsigned long long)hi << 32) | lo);
}

__device__ __forceinline__ void wait_vm(int n) { asm volatile("s_waitcnt vmcnt(%0)" :: "i"(n) : "memory"); }

__device__ __forceinline__ unsigned wait_flag(const unsigned* flag, unsigned want, unsigned* tmo, unsigned call) {
  unsigned v = __builtin_amdgcn_readfirstlane(__hip_atomic_load(flag, __ATOMIC_RELAXED, __HIP_MEMORY_SCOPE_SYSTEM));
  if (v >= want) return v;
  const unsigned long long t0 = __builtin_amdgcn_s_memrealtime();
  for (;;) {
    __builtin_amdgcn_s_sleep(POLL_SLEEP);
    v = __builtin_amdgcn_readfirstlane(__hip_atomic_load(flag, __ATOMIC_RELAXED, __HIP_MEMORY_SCOPE_SYSTEM));
    if (v >= want) return v;
    if (__builtin_amdgcn_readfirstlane(__hip_atomic_load(tmo, __ATOMIC_RELAXED, __HIP_MEMORY_SCOPE_AGENT)) == call ||
        __builtin_amdgcn_s_memrealtime() - t0 > DEADLINE_TICKS) {
      __hip_atomic_store(tmo, call, __ATOMIC_RELAXED, __HIP_MEMORY_SCOPE_AGENT);
      asm volatile("s_waitcnt vmcnt(0)" ::: "memory");
      return 0xffffffffu;
    }
  }
}

#define PUSH_D 1
#define PUSH_TAU 1160
#define PUSH_D2 PUSH_D
#define PUSH_N2 NSTEP
#define PUSH_PP (PUSH_D2 + 3)

__global__ void __launch_bounds__(512, 2) gemm_ag_kernel(Params p) {
  __shared__ __attribute__((aligned(16))) unsigned char lds[LDS_BYTES];
  LAS unsigned char* L = (LAS unsigned char*)lds;
  const unsigned lbase = (unsigned)(size_t)L;

  const int tid = threadIdx.x;
  const int wave = __builtin_amdgcn_readfirstlane(tid >> 6);
  const int lane = tid & 63;
  const int b = blockIdx.x;
  const int xcd = b & 7, idx = b >> 3;
  const int rank = p.rank & (NR - 1);

  STAMP(0);
  const unsigned long long t_entry = __builtin_amdgcn_s_memrealtime();
  unsigned char* ws = p.ws[rank];
  unsigned* call_ctr = (unsigned*)(ws + WS_CALL + (size_t)b * 128);
  unsigned* tmo = (unsigned*)(ws + WS_TMO);
  const unsigned call_raw = __hip_atomic_load(call_ctr, __ATOMIC_RELAXED, __HIP_MEMORY_SCOPE_AGENT);
#define CALL_STATE() \
  const unsigned call = __builtin_amdgcn_readfirstlane(call_raw) + 1u; \
  const unsigned seq0 = call * 64u;   \
  const size_t slot_off = WS_STAGE + (size_t)(call & 1u) * IMG_SLOT

  if (b < 4 * (NR - 1)) {
    if (wave < 4) {
      const int peer = (rank + 1 + b % (NR - 1)) & (NR - 1);
      const int prb = b / (NR - 1);
      const auto rsX = __builtin_amdgcn_make_buffer_rsrc((void*)uni_ptr(p.x + (size_t)(prb * BM + wave * 32) * KT), 0,
                                                         32 * KT, 0x00020000);
      unsigned voff[2];
#pragma unroll
      for (int jj = 0; jj < 2; ++jj) {
        const int m = jj * 8 + (lane >> 3);
        voff[jj] = (unsigned)(m * KT + (((lane & 7) ^ ((m >> 1) & 7)) * 16));
      }
      const unsigned vo = (unsigned)(wave * 4096 + lane * 16);
      constexpr int NP = NSTEP;
      constexpr int PP = PUSH_PP;
      u32x4 d[PP][4];
#define PLOAD(slot, n_) do { const unsigned o_ = (unsigned)(n_) * BK; \
        d[slot][0] = __builtin_amdgcn_raw_buffer_load_b128(rsX, voff[0], o_, 0); \
        d[slot][1] = __builtin_amdgcn_raw_buffer_load_b128(rsX, voff[1], o_, 0); \
        d[slot][2] = __builtin_amdgcn_raw_buffer_load_b128(rsX, voff[0], o_ + 16u * KT, 0); \
        d[slot][3] = __builtin_amdgcn_raw_buffer_load_b128(rsX, voff[1], o_ + 16u * KT, 0); } while (0)
#define PFLAG(n_) do { if (lane == 0) __hip_atomic_store(pflag, seq0 + (unsigned)((n_) + 1), \
        __ATOMIC_RELAXED, __HIP_MEMORY_SCOPE_SYSTEM); } while (0)
      PLOAD(0, 0);
      __builtin_amdgcn_sched_barrier(0);
      CALL_STATE();
      __builtin_amdgcn_sched_barrier(0);
#pragma unroll
      for (int n = 1; n < PP; ++n) PLOAD(n, n);
      unsigned char* pws = p.ws[peer];
      const auto rsP = __builtin_amdgcn_make_buffer_rsrc(
          (void*)uni_ptr(pws + slot_off + (size_t)rank * IMG_SRC + (size_t)prb * IMG_RB), 0, (int)IMG_RB, 0x00020000);
      unsigned* pflag = (unsigned*)(pws + WS_FLAG + (size_t)((rank * NRB + prb) * 4 + wave) * 128);
      STAMP(1);
      const unsigned long long tp0 = __builtin_amdgcn_s_memrealtime();
      unsigned long long tprev = 0;
      (void)tprev;
      int done = 0;
      int nfl[NP + 1];
#pragma unroll
      for (int n = 0; n < NP; ++n) {
        while ((long long)(__builtin_amdgcn_s_memrealtime() - tp0) < (long long)(n * PUSH_TAU / 10)) __builtin_amdgcn_s_sleep(1);
        const unsigned po = (unsigned)n * A_BYTES;
        __builtin_amdgcn_raw_buffer_store_b128(d[n % PP][0], rsP, vo, po, AUX_SYS);
        __builtin_amdgcn_raw_buffer_store_b128(d[n % PP][1], rsP, vo + 1024, po, AUX_SYS);
        __builtin_amdgcn_raw_buffer_store_b128(d[n % PP][2], rsP, vo + 2048, po, AUX_SYS);
        __builtin_amdgcn_raw_buffer_store_b128(d[n % PP][3], rsP, vo + 3072, po, AUX_SYS);
        const int t = n - (n >= PUSH_N2 ? PUSH_D2 : PUSH_D);
        nfl[n] = 0;
        if (t >= done) {
          int c = 4 * (n - t);
#pragma unroll
          for (int q = t; q < n; ++q) c += nfl[q] + (q + PP < NP ? 4 : 0);
          wait_vm(c);
          PFLAG(t);
          nfl[n] = 1;
          done = t + 1;
        }
        if (n + PP < NP) PLOAD(n % PP, n + PP);
      }
      {
        int extra = 0;
#pragma unroll
        for (int t = NP - PUSH_D2; t < NP; ++t) {
          if (t >= done) {
            int c = 4 * (NP - 1 - t) + extra;
#pragma unroll
            for (int q = t; q < NP; ++q) c += nfl[q];
            wait_vm(c);
            PFLAG(t);
            ++extra;
          }
        }
      }
    }
    STAMP(34);
    STAMP(35);
    if (tid == 0) __hip_atomic_store(call_ctr, call_raw + 1u, __ATOMIC_RELAXED, __HIP_MEMORY_SCOPE_AGENT);
    STAMP(36);
    return;
  }

  if (b < 4 * NR) {
    if (tid == 0) __hip_atomic_store(call_ctr, call_raw + 1u, __ATOMIC_RELAXED, __HIP_MEMORY_SCOPE_AGENT);
    return;
  }

  const int srel = 1 + ((idx - 4) >> 2);
  const int cb = (xcd & 1) * 4 + ((idx - 4) & 3);
  const int rb = xcd >> 1;
  const int src = (rank + srel) & (NR - 1);
  const int wm = wave >> 2, wn = wave & 3;
  const int sq = rb * (NR - 1) + (srel - 1);
  const int srowA = sq * 16, srowB = sq < 4 ? 448 + sq * 16 : srowA;
  const bool strip = wm == 0 || sq < 4;
  const float scale = __builtin_bit_cast(float, __builtin_amdgcn_readfirstlane(__builtin_bit_cast(int, p.sx[0] * p.sw[0])));

  unsigned xvoff[2], wvoff[2], svoff;
#pragma unroll
  for (int j = 0; j < 2; ++j) {
    xvoff[j] = (unsigned)(wave * 2048 + j * 1024 + lane * 16);
    const int k = wave * 16 + j * 4 + (lane >> 4);
    wvoff[j] = (unsigned)(k * NT + (((lane & 15) ^ ((k & 7) | (((k >> 4) & 1) << 3))) * 16));
  }
  {
    const int m = (wave & 3) * 4 + ((lane >> 3) & 3);
    svoff = (unsigned)(((wm ? srowB : srowA) + m) * KT + (((lane & 7) ^ ((m >> 1) & 7)) * 16));
  }
  CALL_STATE();
  const auto rsX = __builtin_amdgcn_make_buffer_rsrc(
      (void*)uni_ptr(ws + slot_off + (size_t)src * IMG_SRC + (size_t)rb * IMG_RB), 0, (int)IMG_RB, 0x00020000);
  const auto rsS = __builtin_amdgcn_make_buffer_rsrc((void*)uni_ptr(p.x), 0, MS * KT, 0x00020000);
  const auto rsW = __builtin_amdgcn_make_buffer_rsrc((void*)uni_ptr(p.w + (size_t)rank * NS + (size_t)cb * BN), 0,
                                                     (int)((size_t)KT * NT - (size_t)rank * NS - (size_t)cb * BN), 0x00020000);
  const unsigned* myflag = (const unsigned*)(ws + WS_FLAG + (size_t)((src * NRB + rb) * 4 + (wave >> 1)) * 128);

#define CLAMP(s) ((s) < NSTEP ? (s) : NSTEP - 1)
#define DMAX(so, s, k) __builtin_amdgcn_raw_ptr_buffer_load_lds(rsX, (LAS void*)(L + (so) + wave * 2048 + (k) * 1024), 16, \
    xvoff[k], (unsigned)CLAMP(s) * A_BYTES, 0, X_AUX)
#define DMAW(so, s, k) __builtin_amdgcn_raw_ptr_buffer_load_lds(rsW, (LAS void*)(L + (so) + A_BYTES + wave * 4096 + (k) * 1024), 16, \
    wvoff[(k) & 1], (unsigned)CLAMP(s) * (unsigned)(BK * NT) + (unsigned)((k) >> 1) * (8u * NT), 0, 0)
#define DMAS(so, s) do { if (lane < 32) __builtin_amdgcn_raw_ptr_buffer_load_lds(rsS, (LAS void*)(L + (so) + S_OFF + wave * 512), 16, \
    svoff, (unsigned)CLAMP(s) * BK, 0, 0); } while (0)
  auto load_x = [&](int s) { const unsigned so = (unsigned)((s % NSTAGE) * STAGE_BYTES); DMAX(so, s, 0); DMAX(so, s, 1); };
  auto load_w = [&](int s) { const unsigned so = (unsigned)((s % NSTAGE) * STAGE_BYTES);
    DMAW(so, s, 0); DMAW(so, s, 1); DMAW(so, s, 2); DMAW(so, s, 3); DMAS(so, s); };

  const int fr = lane & 15, g = lane >> 4;
  const unsigned xa0 = lbase + (unsigned)((wm * 64 + fr) * BK + ((g ^ ((fr >> 1) & 7)) * 16));
  const unsigned sa0 = lbase + (unsigned)(S_OFF + (wm * 16 + fr) * BK + ((g ^ ((fr >> 1) & 7)) * 16));
  unsigned wb[4];
  {
    const int q = fr >> 1, ph = fr & 1;
    const int f = q | ((g & 1) << 3);
#pragma unroll
    for (int ni = 0; ni < 4; ++ni)
      wb[ni] = lbase + (unsigned)(A_BYTES + (16 * g + q) * BN + (((wn * 4 + ni) ^ f) * 16) + ph * 8);
  }

  constexpr int ALL = NSTEP + 8;
  int known = 0;
  int xi = 0;
  int pend = 0;
#define UNI(v) v = __builtin_amdgcn_readfirstlane(v)
  LAS unsigned* actl = (LAS unsigned*)(L + NSTAGE * STAGE_BYTES + 32 + wave * 8);
  const unsigned actl_a = lbase + NSTAGE * STAGE_BYTES + 32 + wave * 8;

  auto pre_sync = [&](int j) {
    UNI(xi); UNI(pend);
    if (__builtin_expect(pend == 0 && xi == j + 2, 1)) { WAIT_V(7); return; }
    UNI(known);
    if (pend) {
      WAIT_V(4);
      unsigned pv;
      asm volatile("ds_read_b32 %0, %1\n\ts_waitcnt lgkmcnt(0)" : "=v"(pv) : "v"(actl_a) : "memory");
      const int k = (int)(__builtin_amdgcn_readfirstlane(pv) - seq0);
      known = k > known ? (k < NSTEP ? k : ALL) : known;
      pend = 0;
    } else if (xi > j) {
      WAIT_V(5);
    }
    if (xi <= j) {
      if (known <= j) {
        const unsigned v = wait_flag(myflag, seq0 + (unsigned)(j + 1), tmo, call);
        const int k = (int)(v - seq0);
        known = (v != 0xffffffffu && k < NSTEP) ? k : ALL;
      }
      load_x(j);
      xi = j + 1;
      if (known > j + 1) { load_x(j + 1); xi = j + 2; WAIT_V(2); } else WAIT_V(0);
    } else if (xi == j + 1 && known > j + 1) {
      load_x(j + 1);
      xi = j + 2;
    }
  };
  auto post_x = [&](int j) -> int {
    UNI(known); UNI(xi);
    if (__builtin_expect(xi == j + 2 && known > j + 2, 1)) { xi = j + 3; return 1; }
    return 0;
  };
  auto post_poll = [&](int j) {
    UNI(known);
    if (__builtin_expect(known < j + 6 || j == NSTEP - 1, 0)) {
      const int np = j < NSTEP - 1 ? 1 : 0;
      if (np && lane == 0) __builtin_amdgcn_global_load_lds(myflag, actl, 4, 0, AUX_SYS);
      if (!np && lane == 0) __builtin_amdgcn_global_load_lds(tmo, actl + 1, 4, 0, 16  );
      pend = np;
    }
  };

#define TR8(dst, addr, off) asm volatile("ds_read_b64_tr_b8 %0, %1 offset:%2" : "=v"(dst) : "v"(addr), "i"(off))
#define RD128(dst, addr, off) asm volatile("ds_read_b128 %0, %1 offset:%2" : "=v"(dst) : "v"(addr), "i"(off))
#define TRW(W, addr) do { const unsigned a_ = (addr); TR8(W##a, a_, 0); TR8(W##b, a_, 8 * BN); TR8(W##c, a_, 64 * BN); TR8(W##d, a_, 72 * BN); } while (0)
#define RDX(X, so_) do { const unsigned a_ = xa0 + (so_), b_ = (xa0 ^ 64u) + (so_); \
    RD128(X##0l, a_, 0); RD128(X##0h, b_, 0); RD128(X##1l, a_, 2048); RD128(X##1h, b_, 2048); \
    RD128(X##2l, a_, 4096); RD128(X##2h, b_, 4096); RD128(X##3l, a_, 6144); RD128(X##3h, b_, 6144); \
    RD128(X##4l, sa0 + (so_), 0); RD128(X##4h, (sa0 ^ 64u) + (so_), 0); } while (0)
#define CAT4(W) ((v8i){W##a[0], W##a[1], W##b[0], W##b[1], W##c[0], W##c[1], W##d[0], W##d[1]})
#define CAT2(l, h) ((v8i){l[0], l[1], l[2], l[3], h[0], h[1], h[2], h[3]})
#define MF1(ni, mi, Wv, Xv) asm volatile("v_mfma_f32_16x16x128_f8f6f4 %0, %1, %2, %0 cbsz:1" : "+a"(acc[ni][mi]) : "v"(Wv), "v"(Xv))
#define PINW1(n, W) asm volatile("s_waitcnt lgkmcnt(" #n ")" : "+v"(W##a), "+v"(W##b), "+v"(W##c), "+v"(W##d))
#define PINW2(W, V) asm volatile("s_waitcnt lgkmcnt(0)" : "+v"(W##a), "+v"(W##b), "+v"(W##c), "+v"(W##d), "+v"(V##a), "+v"(V##b), \
    "+v"(V##c), "+v"(V##d) :: "memory")
#define PINX(X, W) asm volatile("s_waitcnt lgkmcnt(0)" : "+v"(W##a), "+v"(W##b), "+v"(W##c), "+v"(W##d), "+v"(X##0l), "+v"(X##0h), \
    "+v"(X##1l), "+v"(X##1h), "+v"(X##2l), "+v"(X##2h), "+v"(X##3l), "+v"(X##3h), "+v"(X##4l), "+v"(X##4h))
#define SB0() __builtin_amdgcn_sched_barrier(0)
#define MFX(ni, mi, W, X) do { SB0(); MF1(ni, mi, CAT4(W), CAT2(X##mi##l, X##mi##h)); SB0(); } while (0)
#define MFS(ni, W, X) do { if (strip) MFX(ni, 4, W, X); } while (0)
#define KHALF1(XC, so_) \
    const unsigned a1_ = wb[1] + (so_), a2_ = wb[2] + (so_), a3_ = wb[3] + (so_); \
    MFX(0, 0, w0, XC); TR8(w1a, a1_, 0); TR8(w1b, a1_, 8 * BN); TR8(w1c, a1_, 64 * BN); \
    MFX(0, 1, w0, XC); TR8(w1d, a1_, 72 * BN); TR8(w2a, a2_, 0); TR8(w2b, a2_, 8 * BN); \
    MFX(0, 2, w0, XC); TR8(w2c, a2_, 64 * BN); TR8(w2d, a2_, 72 * BN); TR8(w3a, a3_, 0); \
    MFX(0, 3, w0, XC); TR8(w3b, a3_, 8 * BN); TR8(w3c, a3_, 64 * BN); TR8(w3d, a3_, 72 * BN); \
    MFS(0, w0, XC); \
    PINW1(8, w1);
#define KSTEP(i, XC, XN, so_, sn_, s2_) do { \
    KHALF1(XC, so_) \
    MFX(1, 0, w1, XC); DMAW(s2_, (i) + 2, 1); \
    MFX(1, 1, w1, XC); DMAW(s2_, (i) + 2, 2); \
    MFX(1, 2, w1, XC); DMAW(s2_, (i) + 2, 3); \
    MFX(1, 3, w1, XC); DMAS(s2_, (i) + 2); \
    MFS(1, w1, XC); \
    pre_sync((i) + 1); \
    PINW2(w2, w3); \
    BAR(); STAMP(3 + (i)); \
    const int nx_ = post_x((i) + 1); \
    const unsigned xl_ = xa0 + (sn_), xh_ = (xa0 ^ 64u) + (sn_), a0_ = wb[0] + (sn_); \
    MFX(2, 0, w2, XC); RD128(XN##0l, xl_, 0); RD128(XN##0h, xh_, 0); TR8(w0a, a0_, 0); \
    MFX(2, 1, w2, XC); TR8(w0b, a0_, 8 * BN); TR8(w0c, a0_, 64 * BN); TR8(w0d, a0_, 72 * BN); \
    MFX(2, 2, w2, XC); RD128(XN##1l, xl_, 2048); RD128(XN##1h, xh_, 2048); \
    MFX(2, 3, w2, XC); RD128(XN##2l, xl_, 4096); RD128(XN##2h, xh_, 4096); \
    MFS(2, w2, XC); \
    MFX(3, 0, w3, XC); RD128(XN##3l, xl_, 6144); RD128(XN##3h, xh_, 6144); \
    MFX(3, 1, w3, XC); RD128(XN##4l, sa0 + (sn_), 0); RD128(XN##4h, (sa0 ^ 64u) + (sn_), 0); \
    if (nx_) { DMAX(so_, (i) + 3, 0); DMAX(so_, (i) + 3, 1); } \
    MFX(3, 2, w3, XC); DMAW(so_, (i) + 3, 0); \
    MFX(3, 3, w3, XC); post_poll((i) + 1); \
    MFS(3, w3, XC); \
    PINX(XN, w0); SB0(); \
  } while (0)
#define KLAST(XC, so_) do { \
    KHALF1(XC, so_) \
    MFX(1, 0, w1, XC); MFX(1, 1, w1, XC); MFX(1, 2, w1, XC); MFX(1, 3, w1, XC); MFS(1, w1, XC); \
    PINW2(w2, w3); \
    MFX(2, 0, w2, XC); MFX(2, 1, w2, XC); MFX(2, 2, w2, XC); MFX(2, 3, w2, XC); MFS(2, w2, XC); \
    MFX(3, 0, w3, XC); MFX(3, 1, w3, XC); MFX(3, 2, w3, XC); MFX(3, 3, w3, XC); MFS(3, w3, XC); \
  } while (0)

  v4f acc[4][5];
  v2i w0a, w0b, w0c, w0d, w1a, w1b, w1c, w1d, w2a, w2b, w2c, w2d, w3a, w3b, w3c, w3d;
  v4i xa0l, xa0h, xa1l, xa1h, xa2l, xa2h, xa3l, xa3h, xa4l, xa4h, xb0l, xb0h, xb1l, xb1h, xb2l, xb2h, xb3l, xb3h, xb4l, xb4h;

  const auto rsO = __builtin_amdgcn_make_buffer_rsrc((void*)uni_ptr(p.out), 0, MT * NS * 4, 0x00020000);
  const unsigned ocol = (unsigned)(cb * BN + wn * 64 + g * 4) * 4u;
  const unsigned ovo = ocol + (unsigned)(src * MS + rb * BM + wm * 64 + fr) * (unsigned)(NS * 4);
  const unsigned svo = ocol + (unsigned)(rank * MS + (wm ? srowB : srowA) + fr) * (unsigned)(NS * 4);
  load_w(0);
  load_w(1);
  STAMP(1);
#pragma unroll
  for (int i = 0; i < 4; ++i)
#pragma unroll
    for (int j = 0; j < 5; ++j) acc[i][j] = (v4f){0.f, 0.f, 0.f, 0.f};
  pre_sync(0);
  BAR();
  STAMP(2);
  CSTAMP(40);
  if (post_x(0)) load_x(2);
  DMAW(2 * STAGE_BYTES, 2, 0);
  post_poll(0);
  RDX(xa, 0u);
  TRW(w0, wb[0]);
  PINX(xa, w0);
  SB0();
  {
    unsigned s0 = 0, s1 = STAGE_BYTES, s2 = 2 * STAGE_BYTES;
    for (int i = 0; i < NSTEP - 2; i += 2) {
      KSTEP(i, xa, xb, s0, s1, s2);
      KSTEP(i + 1, xb, xa, s1, s2, s0);
      const unsigned t_ = s0; s0 = s2; s2 = s1; s1 = t_;
    }
    KSTEP(NSTEP - 2, xa, xb, s0, s1, s2);
    constexpr float LOG2E = 1.4426950408889634f, LIM2 = 60.0f * LOG2E;
    const float nscale2 = -scale * LOG2E;
    float oscale;
#define PINA(ni) asm volatile("" : "+a"(acc[ni][0]), "+a"(acc[ni][1]), "+a"(acc[ni][2]), "+a"(acc[ni][3]), "+a"(acc[ni][4]))
#define EPI1(ni, mi) do { \
      const v4f y = acc[ni][mi] * oscale; \
      const v4f z = acc[ni][mi] * nscale2; \
      v4f d; \
      _Pragma("unroll") for (int r = 0; r < 4; ++r) d[r] = __builtin_amdgcn_exp2f(__builtin_amdgcn_fmed3f(z[r], -LIM2, LIM2)); \
      d = d + 1.0f; \
      _Pragma("unroll") for (int r = 0; r < 4; ++r) d[r] = __builtin_amdgcn_rcpf(d[r]); \
      const v4f o = y * d; \
      __builtin_amdgcn_raw_buffer_store_b128(__builtin_bit_cast(u32x4, o), rsO, \
                                             ((mi) < 4 ? ovo + (unsigned)((mi) * 16 * NS * 4) : svo) + (unsigned)((ni) * 64), 0, OUT_AUX); \
    } while (0)
#define EPI(ni) do { EPI1(ni, 0); EPI1(ni, 1); EPI1(ni, 2); EPI1(ni, 3); if (strip) EPI1(ni, 4); } while (0)
    {
      KHALF1(xb, s1)
      MFX(1, 0, w1, xb); MFX(1, 1, w1, xb); MFX(1, 2, w1, xb); MFX(1, 3, w1, xb); MFS(1, w1, xb);
      PINW2(w2, w3);
      unsigned tv;
      asm volatile("s_waitcnt vmcnt(0)\n\tds_read_b32 %0, %1 offset:4\n\ts_waitcnt lgkmcnt(0)" : "=v"(tv) : "v"(actl_a) : "memory");
      oscale = __builtin_amdgcn_readfirstlane(tv) == call ? __builtin_nanf("") : scale;
      STAMP(34);
      CSTAMP(41);
      MFX(2, 0, w2, xb); MFX(2, 1, w2, xb); MFX(2, 2, w2, xb); MFX(2, 3, w2, xb); MFS(2, w2, xb);
      PINA(0);
      EPI(0);
      MFX(3, 0, w3, xb); MFX(3, 1, w3, xb); MFX(3, 2, w3, xb); MFX(3, 3, w3, xb); MFS(3, w3, xb);
      PINA(1);
      EPI(1);
      SB0();
      PINA(2);
      EPI(2);
      asm volatile("s_nop 15\n\ts_nop 15" ::: "memory");
      PINA(3);
      EPI(3);
      STAMP(35);
    }
  }
  if (tid == 0) __hip_atomic_store(call_ctr, call, __ATOMIC_RELAXED, __HIP_MEMORY_SCOPE_AGENT);
}

}

extern "C" void kernel_launch(int rank, int nranks, ncclComm_t comm,
                              hipStream_t stream,
                              void* const* d_in, const int* in_sizes, int n_in,
                              void* d_out, int out_size,
                              const int* in_dtypes, int out_dtype,
                              void* const* d_out_all,
                              void* const* d_ws_all, size_t ws_bytes) {
  (void)comm; (void)in_dtypes; (void)out_dtype; (void)d_out_all;
  if (nranks != NR || n_in < 4 || (unsigned)rank >= (unsigned)NR || ws_bytes < WS_TOTAL ||
      in_sizes[0] != MS * KT || in_sizes[1] != KT * NT || out_size != MT * NS) {
    fprintf(stderr, "unexpected problem shape\n");
    return;
  }
  Params p;
  p.x = (const unsigned char*)d_in[0];
  p.w = (const unsigned char*)d_in[1];
  p.sx = (const float*)d_in[2];
  p.sw = (const float*)d_in[3];
  p.out = (float*)d_out;
  for (int r = 0; r < NR; ++r) p.ws[r] = (unsigned char*)d_ws_all[r];
  p.rank = rank;
  hipLaunchKernelGGL(gemm_ag_kernel, dim3(NWG), dim3(512), 0, stream, p);
  HIP_CHECK(hipGetLastError());
}
```

```cpp
#include <hip/hip_runtime.h>
#include <rccl/rccl.h>
#include <cstdio>
#include <cstdint>

#define HIP_CHECK(expr) do { hipError_t _e = (expr); if (_e != hipSuccess) { \
    fprintf(stderr, "HIP error %s at %s:%d\n", hipGetErrorString(_e), __FILE__, __LINE__); \
    return; } } while (0)

namespace {

#define LAS __attribute__((address_space(3)))
typedef int v2i __attribute__((ext_vector_type(2)));
typedef int v4i __attribute__((ext_vector_type(4)));
typedef int v8i __attribute__((ext_vector_type(8)));
typedef unsigned u32x4 __attribute__((ext_vector_type(4)));
typedef float v4f __attribute__((ext_vector_type(4)));

constexpr int NR = 8;
constexpr int MT = 4096, KT = 4096, NT = 16384;
constexpr int MS = MT / NR;
constexpr int NS = NT / NR;
constexpr int BM = 128, BN = 256, BK = 128;
constexpr int NSTEP = KT / BK;
constexpr int NRB = MS / BM;
constexpr int NCB = NS / BN;
constexpr int NWG = (MT / BM) * NCB;
constexpr int A_BYTES = BM * BK;
constexpr int B_BYTES = BK * BN;
constexpr int S_BYTES = 2 * 16 * BK;
constexpr int S_OFF = A_BYTES + B_BYTES;
constexpr int STAGE_BYTES = A_BYTES + B_BYTES + S_BYTES;
constexpr int NSTAGE = 3;
constexpr int LDS_BYTES = NSTAGE * STAGE_BYTES + 128;

constexpr size_t WS_CALL = 0;
constexpr size_t WS_TMO = WS_CALL + (size_t)NWG * 128;
constexpr size_t WS_FLAG = WS_TMO + 128;
constexpr size_t WS_STAGE = 1u << 20;
constexpr size_t IMG_RB = (size_t)NSTEP * A_BYTES;
constexpr size_t IMG_SRC = IMG_RB * NRB;
constexpr size_t IMG_SLOT = IMG_SRC * NR;
constexpr size_t WS_TOTAL = WS_STAGE + 2 * IMG_SLOT;
static_assert(WS_FLAG + (size_t)NR * NRB * 4 * 128 <= WS_STAGE, "control block");

constexpr unsigned long long DEADLINE_TICKS = 40000000ull;
constexpr int AUX_WT = 16;
constexpr int AUX_SYS = 17;

struct Params {
  const unsigned char* x;
  const unsigned char* w;
  const float* sx;
  const float* sw;
  float* out;
  unsigned char* ws[NR];
  int rank;
};

#define WAIT_V(n) asm volatile("s_waitcnt vmcnt(" #n ")" ::: "memory")
#define BAR() do { asm volatile("" ::: "memory"); __builtin_amdgcn_s_barrier(); asm volatile("" ::: "memory"); } while (0)

__device__ __forceinline__ const void* uni_ptr(const void* p) {
  const unsigned long long v = (unsigned long long)p;
  const unsigned lo = __builtin_amdgcn_readfirstlane((unsigned)v);
  const unsigned hi = __builtin_amdgcn_readfirstlane((unsigned)(v >> 32));
  return (const void*)(((unsigned long long)hi << 32) | lo);
}

__device__ __forceinline__ unsigned wait_flag(const unsigned* flag, unsigned want, unsigned* tmo, unsigned call) {
  unsigned v = __builtin_amdgcn_readfirstlane(__hip_atomic_load(flag, __ATOMIC_RELAXED, __HIP_MEMORY_SCOPE_SYSTEM));
  if (v >= want) return v;
  const unsigned long long t0 = __builtin_amdgcn_s_memrealtime();
  for (;;) {
    __builtin_amdgcn_s_sleep(1);
    v = __builtin_amdgcn_readfirstlane(__hip_atomic_load(flag, __ATOMIC_RELAXED, __HIP_MEMORY_SCOPE_SYSTEM));
    if (v >= want) return v;
    if (__builtin_amdgcn_readfirstlane(__hip_atomic_load(tmo, __ATOMIC_RELAXED, __HIP_MEMORY_SCOPE_AGENT)) == call ||
        __builtin_amdgcn_s_memrealtime() - t0 > DEADLINE_TICKS) {
      __hip_atomic_store(tmo, call, __ATOMIC_RELAXED, __HIP_MEMORY_SCOPE_AGENT);
      asm volatile("s_waitcnt vmcnt(0)" ::: "memory");
      return 0xffffffffu;
    }
  }
}

constexpr int PUSH_TICKS = 111;
constexpr int PP = 4;

__global__ void __launch_bounds__(512, 2) gemm_ag_kernel(Params p) {
  __shared__ __attribute__((aligned(16))) unsigned char lds[LDS_BYTES];
  LAS unsigned char* L = (LAS unsigned char*)lds;
  const unsigned lbase = (unsigned)(size_t)L;

  const int tid = threadIdx.x;
  const int wave = __builtin_amdgcn_readfirstlane(tid >> 6);
  const int lane = tid & 63;
  const int b = blockIdx.x;
  const int xcd = b & 7, idx = b >> 3;
  const int rank = p.rank & (NR - 1);
  unsigned char* ws = p.ws[rank];
  unsigned* call_ctr = (unsigned*)(ws + WS_CALL + (size_t)b * 128);
  unsigned* tmo = (unsigned*)(ws + WS_TMO);
  const unsigned call_raw = __hip_atomic_load(call_ctr, __ATOMIC_RELAXED, __HIP_MEMORY_SCOPE_AGENT);
#define CALL_STATE() \
  const unsigned call = __builtin_amdgcn_readfirstlane(call_raw) + 1u; \
  const unsigned seq0 = call * 64u;   \
  const size_t slot_off = WS_STAGE + (size_t)(call & 1u) * IMG_SLOT

  if (b < 4 * (NR - 1)) {
    if (wave < 4) {
      const int peer = (rank + 1 + b % (NR - 1)) & (NR - 1);
      const int prb = b / (NR - 1);
      const auto rsX = __builtin_amdgcn_make_buffer_rsrc((void*)uni_ptr(p.x + (size_t)(prb * BM + wave * 32) * KT), 0,
                                                         32 * KT, 0x00020000);
      unsigned voff[2];
#pragma unroll
      for (int jj = 0; jj < 2; ++jj) {
        const int m = jj * 8 + (lane >> 3);
        voff[jj] = (unsigned)(m * KT + (((lane & 7) ^ ((m >> 1) & 7)) * 16));
      }
      const unsigned vo = (unsigned)(wave * 4096 + lane * 16);
      u32x4 d[PP][4];
#define PLOAD(slot, n_) do { const unsigned o_ = (unsigned)(n_) * BK; \
        d[slot][0] = __builtin_amdgcn_raw_buffer_load_b128(rsX, voff[0], o_, 0); \
        d[slot][1] = __builtin_amdgcn_raw_buffer_load_b128(rsX, voff[1], o_, 0); \
        d[slot][2] = __builtin_amdgcn_raw_buffer_load_b128(rsX, voff[0], o_ + 16u * KT, 0); \
        d[slot][3] = __builtin_amdgcn_raw_buffer_load_b128(rsX, voff[1], o_ + 16u * KT, 0); } while (0)
      PLOAD(0, 0);
      __builtin_amdgcn_sched_barrier(0);
      CALL_STATE();
      __builtin_amdgcn_sched_barrier(0);
#pragma unroll
      for (int n = 1; n < PP; ++n) PLOAD(n, n);
      unsigned char* pws = p.ws[peer];
      const auto rsP = __builtin_amdgcn_make_buffer_rsrc(
          (void*)uni_ptr(pws + slot_off + (size_t)rank * IMG_SRC + (size_t)prb * IMG_RB), 0, (int)IMG_RB, 0x00020000);
      unsigned* pflag = (unsigned*)(pws + WS_FLAG + (size_t)((rank * NRB + prb) * 4 + wave) * 128);
      const unsigned long long tp0 = __builtin_amdgcn_s_memrealtime();
      for (int n0 = 0; n0 < NSTEP; n0 += PP) {
#pragma unroll
        for (int q = 0; q < PP; ++q) {
          const int n = n0 + q;
          while (__builtin_amdgcn_s_memrealtime() - tp0 < (unsigned long long)(n * PUSH_TICKS)) __builtin_amdgcn_s_sleep(1);
          const unsigned po = (unsigned)n * A_BYTES;
          __builtin_amdgcn_raw_buffer_store_b128(d[q][0], rsP, vo, po, AUX_SYS);
          __builtin_amdgcn_raw_buffer_store_b128(d[q][1], rsP, vo + 1024, po, AUX_SYS);
          __builtin_amdgcn_raw_buffer_store_b128(d[q][2], rsP, vo + 2048, po, AUX_SYS);
          __builtin_amdgcn_raw_buffer_store_b128(d[q][3], rsP, vo + 3072, po, AUX_SYS);
          WAIT_V(0);
          if (lane == 0) __hip_atomic_store(pflag, seq0 + (unsigned)(n + 1), __ATOMIC_RELAXED, __HIP_MEMORY_SCOPE_SYSTEM);
          if (n + PP < NSTEP) PLOAD(q, n + PP);
        }
      }
    }
    if (tid == 0) __hip_atomic_store(call_ctr, call_raw + 1u, __ATOMIC_RELAXED, __HIP_MEMORY_SCOPE_AGENT);
    return;
  }

  if (b < 4 * NR) {
    if (tid == 0) __hip_atomic_store(call_ctr, call_raw + 1u, __ATOMIC_RELAXED, __HIP_MEMORY_SCOPE_AGENT);
    return;
  }

  const int srel = 1 + ((idx - 4) >> 2);
  const int cb = (xcd & 1) * 4 + ((idx - 4) & 3);
  const int rb = xcd >> 1;
  const int src = (rank + srel) & (NR - 1);
  const int wm = wave >> 2, wn = wave & 3;
  const int sq = rb * (NR - 1) + (srel - 1);
  const int srowA = sq * 16, srowB = sq < 4 ? 448 + sq * 16 : srowA;
  const bool strip = wm == 0 || sq < 4;
  const float scale = __builtin_bit_cast(float, __builtin_amdgcn_readfirstlane(__builtin_bit_cast(int, p.sx[0] * p.sw[0])));

  unsigned xvoff[2], wvoff[2], svoff;
#pragma unroll
  for (int j = 0; j < 2; ++j) {
    xvoff[j] = (unsigned)(wave * 2048 + j * 1024 + lane * 16);
    const int k = wave * 16 + j * 4 + (lane >> 4);
    wvoff[j] = (unsigned)(k * NT + (((lane & 15) ^ ((k & 7) | (((k >> 4) & 1) << 3))) * 16));
  }
  {
    const int m = (wave & 3) * 4 + ((lane >> 3) & 3);
    svoff = (unsigned)(((wm ? srowB : srowA) + m) * KT + (((lane & 7) ^ ((m >> 1) & 7)) * 16));
  }
  CALL_STATE();
  const auto rsX = __builtin_amdgcn_make_buffer_rsrc(
      (void*)uni_ptr(ws + slot_off + (size_t)src * IMG_SRC + (size_t)rb * IMG_RB), 0, (int)IMG_RB, 0x00020000);
  const auto rsS = __builtin_amdgcn_make_buffer_rsrc((void*)uni_ptr(p.x), 0, MS * KT, 0x00020000);
  const auto rsW = __builtin_amdgcn_make_buffer_rsrc((void*)uni_ptr(p.w + (size_t)rank * NS + (size_t)cb * BN), 0,
                                                     (int)((size_t)KT * NT - (size_t)rank * NS - (size_t)cb * BN), 0x00020000);
  const unsigned* myflag = (const unsigned*)(ws + WS_FLAG + (size_t)((src * NRB + rb) * 4 + (wave >> 1)) * 128);

#define CLAMP(s) ((s) < NSTEP ? (s) : NSTEP - 1)
#define DMAX(so, s, k) __builtin_amdgcn_raw_ptr_buffer_load_lds(rsX, (LAS void*)(L + (so) + wave * 2048 + (k) * 1024), 16, \
    xvoff[k], (unsigned)CLAMP(s) * A_BYTES, 0, AUX_SYS)
#define DMAW(so, s, k) __builtin_amdgcn_raw_ptr_buffer_load_lds(rsW, (LAS void*)(L + (so) + A_BYTES + wave * 4096 + (k) * 1024), 16, \
    wvoff[(k) & 1], (unsigned)CLAMP(s) * (unsigned)(BK * NT) + (unsigned)((k) >> 1) * (8u * NT), 0, 0)
#define DMAS(so, s) do { if (lane < 32) __builtin_amdgcn_raw_ptr_buffer_load_lds(rsS, (LAS void*)(L + (so) + S_OFF + wave * 512), 16, \
    svoff, (unsigned)CLAMP(s) * BK, 0, 0); } while (0)
  auto load_x = [&](int s) { const unsigned so = (unsigned)((s % NSTAGE) * STAGE_BYTES); DMAX(so, s, 0); DMAX(so, s, 1); };
  auto load_w = [&](int s) { const unsigned so = (unsigned)((s % NSTAGE) * STAGE_BYTES);
    DMAW(so, s, 0); DMAW(so, s, 1); DMAW(so, s, 2); DMAW(so, s, 3); DMAS(so, s); };

  const int fr = lane & 15, g = lane >> 4;
  const unsigned xa0 = lbase + (unsigned)((wm * 64 + fr) * BK + ((g ^ ((fr >> 1) & 7)) * 16));
  const unsigned sa0 = lbase + (unsigned)(S_OFF + (wm * 16 + fr) * BK + ((g ^ ((fr >> 1) & 7)) * 16));
  unsigned wb[4];
  {
    const int q = fr >> 1, ph = fr & 1;
    const int f = q | ((g & 1) << 3);
#pragma unroll
    for (int ni = 0; ni < 4; ++ni)
      wb[ni] = lbase + (unsigned)(A_BYTES + (16 * g + q) * BN + (((wn * 4 + ni) ^ f) * 16) + ph * 8);
  }

  constexpr int ALL = NSTEP + 8;
  int known = 0;
  int xi = 0;
  int pend = 0;
#define UNI(v) v = __builtin_amdgcn_readfirstlane(v)
  LAS unsigned* actl = (LAS unsigned*)(L + NSTAGE * STAGE_BYTES + 32 + wave * 8);
  const unsigned actl_a = lbase + NSTAGE * STAGE_BYTES + 32 + wave * 8;

  auto pre_sync = [&](int j) {
    UNI(xi); UNI(pend);
    if (__builtin_expect(pend == 0 && xi == j + 2, 1)) { WAIT_V(7); return; }
    UNI(known);
    if (pend) {
      WAIT_V(4);
      unsigned pv;
      asm volatile("ds_read_b32 %0, %1\n\ts_waitcnt lgkmcnt(0)" : "=v"(pv) : "v"(actl_a) : "memory");
      const int k = (int)(__builtin_amdgcn_readfirstlane(pv) - seq0);
      known = k > known ? (k < NSTEP ? k : ALL) : known;
      pend = 0;
    } else if (xi > j) {
      WAIT_V(5);
    }
    if (xi <= j) {
      if (known <= j) {
        const unsigned v = wait_flag(myflag, seq0 + (unsigned)(j + 1), tmo, call);
        const int k = (int)(v - seq0);
        known = (v != 0xffffffffu && k < NSTEP) ? k : ALL;
      }
      load_x(j);
      xi = j + 1;
      if (known > j + 1) { load_x(j + 1); xi = j + 2; WAIT_V(2); } else WAIT_V(0);
    } else if (xi == j + 1 && known > j + 1) {
      load_x(j + 1);
      xi = j + 2;
    }
  };
  auto post_x = [&](int j) -> int {
    UNI(known); UNI(xi);
    if (__builtin_expect(xi == j + 2 && known > j + 2, 1)) { xi = j + 3; return 1; }
    return 0;
  };
  auto post_poll = [&](int j) {
    UNI(known);
    if (__builtin_expect(known < j + 6 || j == NSTEP - 1, 0)) {
      const int np = j < NSTEP - 1 ? 1 : 0;
      if (np && lane == 0) __builtin_amdgcn_global_load_lds(myflag, actl, 4, 0, AUX_SYS);
      if (!np && lane == 0) __builtin_amdgcn_global_load_lds(tmo, actl + 1, 4, 0, 16  );
      pend = np;
    }
  };

#define TR8(dst, addr, off) asm volatile("ds_read_b64_tr_b8 %0, %1 offset:%2" : "=v"(dst) : "v"(addr), "i"(off))
#define RD128(dst, addr, off) asm volatile("ds_read_b128 %0, %1 offset:%2" : "=v"(dst) : "v"(addr), "i"(off))
#define TRW(W, addr) do { const unsigned a_ = (addr); TR8(W##a, a_, 0); TR8(W##b, a_, 8 * BN); TR8(W##c, a_, 64 * BN); TR8(W##d, a_, 72 * BN); } while (0)
#define RDX(X, so_) do { const unsigned a_ = xa0 + (so_), b_ = (xa0 ^ 64u) + (so_); \
    RD128(X##0l, a_, 0); RD128(X##0h, b_, 0); RD128(X##1l, a_, 2048); RD128(X##1h, b_, 2048); \
    RD128(X##2l, a_, 4096); RD128(X##2h, b_, 4096); RD128(X##3l, a_, 6144); RD128(X##3h, b_, 6144); \
    RD128(X##4l, sa0 + (so_), 0); RD128(X##4h, (sa0 ^ 64u) + (so_), 0); } while (0)
#define CAT4(W) ((v8i){W##a[0], W##a[1], W##b[0], W##b[1], W##c[0], W##c[1], W##d[0], W##d[1]})
#define CAT2(l, h) ((v8i){l[0], l[1], l[2], l[3], h[0], h[1], h[2], h[3]})
#define MF1(ni, mi, Wv, Xv) asm volatile("v_mfma_f32_16x16x128_f8f6f4 %0, %1, %2, %0 cbsz:1" : "+a"(acc[ni][mi]) : "v"(Wv), "v"(Xv))
#define PINW1(n, W) asm volatile("s_waitcnt lgkmcnt(" #n ")" : "+v"(W##a), "+v"(W##b), "+v"(W##c), "+v"(W##d))
#define PINW2(W, V) asm volatile("s_waitcnt lgkmcnt(0)" : "+v"(W##a), "+v"(W##b), "+v"(W##c), "+v"(W##d), "+v"(V##a), "+v"(V##b), \
    "+v"(V##c), "+v"(V##d) :: "memory")
#define PINX(X, W) asm volatile("s_waitcnt lgkmcnt(0)" : "+v"(W##a), "+v"(W##b), "+v"(W##c), "+v"(W##d), "+v"(X##0l), "+v"(X##0h), \
    "+v"(X##1l), "+v"(X##1h), "+v"(X##2l), "+v"(X##2h), "+v"(X##3l), "+v"(X##3h), "+v"(X##4l), "+v"(X##4h))
#define SB0() __builtin_amdgcn_sched_barrier(0)
#define MFX(ni, mi, W, X) do { SB0(); MF1(ni, mi, CAT4(W), CAT2(X##mi##l, X##mi##h)); SB0(); } while (0)
#define MFS(ni, W, X) do { if (strip) MFX(ni, 4, W, X); } while (0)
#define KHALF1(XC, so_) \
    const unsigned a1_ = wb[1] + (so_), a2_ = wb[2] + (so_), a3_ = wb[3] + (so_); \
    MFX(0, 0, w0, XC); TR8(w1a, a1_, 0); TR8(w1b, a1_, 8 * BN); TR8(w1c, a1_, 64 * BN); \
    MFX(0, 1, w0, XC); TR8(w1d, a1_, 72 * BN); TR8(w2a, a2_, 0); TR8(w2b, a2_, 8 * BN); \
    MFX(0, 2, w0, XC); TR8(w2c, a2_, 64 * BN); TR8(w2d, a2_, 72 * BN); TR8(w3a, a3_, 0); \
    MFX(0, 3, w0, XC); TR8(w3b, a3_, 8 * BN); TR8(w3c, a3_, 64 * BN); TR8(w3d, a3_, 72 * BN); \
    MFS(0, w0, XC); \
    PINW1(8, w1);
#define KSTEP(i, XC, XN, so_, sn_, s2_) do { \
    KHALF1(XC, so_) \
    MFX(1, 0, w1, XC); DMAW(s2_, (i) + 2, 1); \
    MFX(1, 1, w1, XC); DMAW(s2_, (i) + 2, 2); \
    MFX(1, 2, w1, XC); DMAW(s2_, (i) + 2, 3); \
    MFX(1, 3, w1, XC); DMAS(s2_, (i) + 2); \
    MFS(1, w1, XC); \
    pre_sync((i) + 1); \
    PINW2(w2, w3); \
    BAR(); \
    const int nx_ = post_x((i) + 1); \
    const unsigned xl_ = xa0 + (sn_), xh_ = (xa0 ^ 64u) + (sn_), a0_ = wb[0] + (sn_); \
    MFX(2, 0, w2, XC); RD128(XN##0l, xl_, 0); RD128(XN##0h, xh_, 0); TR8(w0a, a0_, 0); \
    MFX(2, 1, w2, XC); TR8(w0b, a0_, 8 * BN); TR8(w0c, a0_, 64 * BN); TR8(w0d, a0_, 72 * BN); \
    MFX(2, 2, w2, XC); RD128(XN##1l, xl_, 2048); RD128(XN##1h, xh_, 2048); \
    MFX(2, 3, w2, XC); RD128(XN##2l, xl_, 4096); RD128(XN##2h, xh_, 4096); \
    MFS(2, w2, XC); \
    MFX(3, 0, w3, XC); RD128(XN##3l, xl_, 6144); RD128(XN##3h, xh_, 6144); \
    MFX(3, 1, w3, XC); RD128(XN##4l, sa0 + (sn_), 0); RD128(XN##4h, (sa0 ^ 64u) + (sn_), 0); \
    if (nx_) { DMAX(so_, (i) + 3, 0); DMAX(so_, (i) + 3, 1); } \
    MFX(3, 2, w3, XC); DMAW(so_, (i) + 3, 0); \
    MFX(3, 3, w3, XC); post_poll((i) + 1); \
    MFS(3, w3, XC); \
    PINX(XN, w0); SB0(); \
  } while (0)
#define KLAST(XC, so_) do { \
    KHALF1(XC, so_) \
    MFX(1, 0, w1, XC); MFX(1, 1, w1, XC); MFX(1, 2, w1, XC); MFX(1, 3, w1, XC); MFS(1, w1, XC); \
    PINW2(w2, w3); \
    MFX(2, 0, w2, XC); MFX(2, 1, w2, XC); MFX(2, 2, w2, XC); MFX(2, 3, w2, XC); MFS(2, w2, XC); \
    MFX(3, 0, w3, XC); MFX(3, 1, w3, XC); MFX(3, 2, w3, XC); MFX(3, 3, w3, XC); MFS(3, w3, XC); \
  } while (0)

  v4f acc[4][5];
  v2i w0a, w0b, w0c, w0d, w1a, w1b, w1c, w1d, w2a, w2b, w2c, w2d, w3a, w3b, w3c, w3d;
  v4i xa0l, xa0h, xa1l, xa1h, xa2l, xa2h, xa3l, xa3h, xa4l, xa4h, xb0l, xb0h, xb1l, xb1h, xb2l, xb2h, xb3l, xb3h, xb4l, xb4h;

  const auto rsO = __builtin_amdgcn_make_buffer_rsrc((void*)uni_ptr(p.out), 0, MT * NS * 4, 0x00020000);
  const unsigned ocol = (unsigned)(cb * BN + wn * 64 + g * 4) * 4u;
  const unsigned ovo = ocol + (unsigned)(src * MS + rb * BM + wm * 64 + fr) * (unsigned)(NS * 4);
  const unsigned svo = ocol + (unsigned)(rank * MS + (wm ? srowB : srowA) + fr) * (unsigned)(NS * 4);
  load_w(0);
  load_w(1);
#pragma unroll
  for (int i = 0; i < 4; ++i)
#pragma unroll
    for (int j = 0; j < 5; ++j) acc[i][j] = (v4f){0.f, 0.f, 0.f, 0.f};
  pre_sync(0);
  BAR();
  if (post_x(0)) load_x(2);
  DMAW(2 * STAGE_BYTES, 2, 0);
  post_poll(0);
  RDX(xa, 0u);
  TRW(w0, wb[0]);
  PINX(xa, w0);
  SB0();
  {
    unsigned s0 = 0, s1 = STAGE_BYTES, s2 = 2 * STAGE_BYTES;
    for (int i = 0; i < NSTEP - 2; i += 2) {
      KSTEP(i, xa, xb, s0, s1, s2);
      KSTEP(i + 1, xb, xa, s1, s2, s0);
      const unsigned t_ = s0; s0 = s2; s2 = s1; s1 = t_;
    }
    KSTEP(NSTEP - 2, xa, xb, s0, s1, s2);
    constexpr float LOG2E = 1.4426950408889634f, LIM2 = 60.0f * LOG2E;
    const float nscale2 = -scale * LOG2E;
    float oscale;
#define PINA(ni) asm volatile("" : "+a"(acc[ni][0]), "+a"(acc[ni][1]), "+a"(acc[ni][2]), "+a"(acc[ni][3]), "+a"(acc[ni][4]))
#define EPI1(ni, mi) do { \
      const v4f y = acc[ni][mi] * oscale; \
      const v4f z = acc[ni][mi] * nscale2; \
      v4f d; \
      _Pragma("unroll") for (int r = 0; r < 4; ++r) d[r] = __builtin_amdgcn_exp2f(__builtin_amdgcn_fmed3f(z[r], -LIM2, LIM2)); \
      d = d + 1.0f; \
      _Pragma("unroll") for (int r = 0; r < 4; ++r) d[r] = __builtin_amdgcn_rcpf(d[r]); \
      const v4f o = y * d; \
      __builtin_amdgcn_raw_buffer_store_b128(__builtin_bit_cast(u32x4, o), rsO, \
                                             ((mi) < 4 ? ovo + (unsigned)((mi) * 16 * NS * 4) : svo) + (unsigned)((ni) * 64), 0, AUX_WT); \
    } while (0)
#define EPI(ni) do { EPI1(ni, 0); EPI1(ni, 1); EPI1(ni, 2); EPI1(ni, 3); if (strip) EPI1(ni, 4); } while (0)
    {
      KHALF1(xb, s1)
      MFX(1, 0, w1, xb); MFX(1, 1, w1, xb); MFX(1, 2, w1, xb); MFX(1, 3, w1, xb); MFS(1, w1, xb);
      PINW2(w2, w3);
      unsigned tv;
      asm volatile("s_waitcnt vmcnt(0)\n\tds_read_b32 %0, %1 offset:4\n\ts_waitcnt lgkmcnt(0)" : "=v"(tv) : "v"(actl_a) : "memory");
      oscale = __builtin_amdgcn_readfirstlane(tv) == call ? __builtin_nanf("") : scale;
      MFX(2, 0, w2, xb); MFX(2, 1, w2, xb); MFX(2, 2, w2, xb); MFX(2, 3, w2, xb); MFS(2, w2, xb);
      PINA(0);
      EPI(0);
      MFX(3, 0, w3, xb); MFX(3, 1, w3, xb); MFX(3, 2, w3, xb); MFX(3, 3, w3, xb); MFS(3, w3, xb);
      PINA(1);
      EPI(1);
      SB0();
      PINA(2);
      EPI(2);
      asm volatile("s_nop 15\n\ts_nop 15" ::: "memory");
      PINA(3);
      EPI(3);
    }
  }
  if (tid == 0) __hip_atomic_store(call_ctr, call, __ATOMIC_RELAXED, __HIP_MEMORY_SCOPE_AGENT);
}

}

extern "C" void kernel_launch(int rank, int nranks, ncclComm_t comm,
                              hipStream_t stream,
                              void* const* d_in, const int* in_sizes, int n_in,
                              void* d_out, int out_size,
                              const int* in_dtypes, int out_dtype,
                              void* const* d_out_all,
                              void* const* d_ws_all, size_t ws_bytes) {
  (void)comm; (void)in_dtypes; (void)out_dtype; (void)d_out_all;
  if (nranks != NR || n_in < 4 || (unsigned)rank >= (unsigned)NR || ws_bytes < WS_TOTAL ||
      in_sizes[0] != MS * KT || in_sizes[1] != KT * NT || out_size != MT * NS) {
    fprintf(stderr, "unexpected problem shape\n");
    return;
  }
  Params p;
  p.x = (const unsigned char*)d_in[0];
  p.w = (const unsigned char*)d_in[1];
  p.sx = (const float*)d_in[2];
  p.sw = (const float*)d_in[3];
  p.out = (float*)d_out;
  for (int r = 0; r < NR; ++r) p.ws[r] = (unsigned char*)d_ws_all[r];
  p.rank = rank;
  hipLaunchKernelGGL(gemm_ag_kernel, dim3(NWG), dim3(512), 0, stream, p);
  HIP_CHECK(hipGetLastError());
}
```

```cpp
#include <hip/hip_runtime.h>
#include <rccl/rccl.h>
#include <cstdio>
#include <cstdint>

#define HIP_CHECK(expr) do { hipError_t _e = (expr); if (_e != hipSuccess) { \
    fprintf(stderr, "HIP error %s at %s:%d\n", hipGetErrorString(_e), __FILE__, __LINE__); \
    return; } } while (0)

namespace {

#define LAS __attribute__((address_space(3)))
typedef int v2i __attribute__((ext_vector_type(2)));
typedef int v4i __attribute__((ext_vector_type(4)));
typedef int v8i __attribute__((ext_vector_type(8)));
typedef unsigned u32x4 __attribute__((ext_vector_type(4)));
typedef float v4f __attribute__((ext_vector_type(4)));

constexpr int NR = 8;
constexpr int MT = 4096, KT = 4096, NT = 16384;
constexpr int MS = MT / NR;
constexpr int NS = NT / NR;
constexpr int BM = 128, BN = 256, BK = 128;
constexpr int NSTEP = KT / BK;
constexpr int NRB = MS / BM;
constexpr int NCB = NS / BN;
constexpr int NWG = (MT / BM) * NCB;
constexpr int A_BYTES = BM * BK;
constexpr int B_BYTES = BK * BN;
constexpr int S_BYTES = 2 * 16 * BK;
constexpr int S_OFF = A_BYTES + B_BYTES;
constexpr int STAGE_BYTES = A_BYTES + B_BYTES + S_BYTES;
constexpr int NSTAGE = 3;
constexpr int LDS_BYTES = NSTAGE * STAGE_BYTES + 128;

constexpr size_t WS_CALL = 0;
constexpr size_t WS_TMO = WS_CALL + (size_t)NWG * 128;
constexpr size_t WS_FLAG = WS_TMO + 128;
constexpr size_t WS_STAGE = 1u << 20;
constexpr size_t IMG_RB = (size_t)NSTEP * A_BYTES;
constexpr size_t IMG_SRC = IMG_RB * NRB;
constexpr size_t IMG_SLOT = IMG_SRC * NR;
constexpr size_t WS_TOTAL = WS_STAGE + 2 * IMG_SLOT;
static_assert(WS_FLAG + (size_t)NR * NRB * 4 * 128 <= WS_STAGE, "control block");

constexpr unsigned long long DEADLINE_TICKS = 40000000ull;
constexpr int AUX_WT = 16;
constexpr int AUX_SYS = 17;

struct Params {
  const unsigned char* x;
  const unsigned char* w;
  const float* sx;
  const float* sw;
  float* out;
  unsigned char* ws[NR];
  int rank;
};

#define WAIT_V(n) asm volatile("s_waitcnt vmcnt(" #n ")" ::: "memory")
#define BAR() do { asm volatile("" ::: "memory"); __builtin_amdgcn_s_barrier(); asm volatile("" ::: "memory"); } while (0)

__device__ __forceinline__ const void* uni_ptr(const void* p) {
  const unsigned long long v = (unsigned long long)p;
  const unsigned lo = __builtin_amdgcn_readfirstlane((unsigned)v);
  const unsigned hi = __builtin_amdgcn_readfirstlane((unsigned)(v >> 32));
  return (const void*)(((unsigned long long)hi << 32) | lo);
}

__device__ __forceinline__ unsigned wait_flag(const unsigned* flag, unsigned want, unsigned* tmo, unsigned call) {
  unsigned v = __builtin_amdgcn_readfirstlane(__hip_atomic_load(flag, __ATOMIC_RELAXED, __HIP_MEMORY_SCOPE_SYSTEM));
  if (v >= want) return v;
  const unsigned long long t0 = __builtin_amdgcn_s_memrealtime();
  for (;;) {
    __builtin_amdgcn_s_sleep(1);
    v = __builtin_amdgcn_readfirstlane(__hip_atomic_load(flag, __ATOMIC_RELAXED, __HIP_MEMORY_SCOPE_SYSTEM));
    if (v >= want) return v;
    if (__builtin_amdgcn_readfirstlane(__hip_atomic_load(tmo, __ATOMIC_RELAXED, __HIP_MEMORY_SCOPE_AGENT)) == call ||
        __builtin_amdgcn_s_memrealtime() - t0 > DEADLINE_TICKS) {
      __hip_atomic_store(tmo, call, __ATOMIC_RELAXED, __HIP_MEMORY_SCOPE_AGENT);
      asm volatile("s_waitcnt vmcnt(0)" ::: "memory");
      return 0xffffffffu;
    }
  }
}

constexpr int PUSH_TICKS = 111;
constexpr int PP = 4;

__global__ void __launch_bounds__(512, 2) gemm_ag_kernel(Params p) {
  __shared__ __attribute__((aligned(16))) unsigned char lds[LDS_BYTES];
  LAS unsigned char* L = (LAS unsigned char*)lds;
  const unsigned lbase = (unsigned)(size_t)L;

  const int tid = threadIdx.x;
  const int wave = __builtin_amdgcn_readfirstlane(tid >> 6);
  const int lane = tid & 63;
  const int b = blockIdx.x;
  const int xcd = b & 7, idx = b >> 3;
  const int rank = p.rank & (NR - 1);
  unsigned char* ws = p.ws[rank];
  unsigned* call_ctr = (unsigned*)(ws + WS_CALL + (size_t)b * 128);
  unsigned* tmo = (unsigned*)(ws + WS_TMO);
  const unsigned call_raw = __hip_atomic_load(call_ctr, __ATOMIC_RELAXED, __HIP_MEMORY_SCOPE_AGENT);
#define CALL_STATE() \
  const unsigned call = __builtin_amdgcn_readfirstlane(call_raw) + 1u; \
  const unsigned seq0 = call * 64u;   \
  const size_t slot_off = WS_STAGE + (size_t)(call & 1u) * IMG_SLOT

  if (b < 4 * (NR - 1)) {
    if (wave < 4) {
      const int peer = (rank + 1 + b % (NR - 1)) & (NR - 1);
      const int prb = b / (NR - 1);
      const auto rsX = __builtin_amdgcn_make_buffer_rsrc((void*)uni_ptr(p.x + (size_t)(prb * BM + wave * 32) * KT), 0,
                                                         32 * KT, 0x00020000);
      unsigned voff[2];
#pragma unroll
      for (int jj = 0; jj < 2; ++jj) {
        const int m = jj * 8 + (lane >> 3);
        voff[jj] = (unsigned)(m * KT + (((lane & 7) ^ ((m >> 1) & 7)) * 16));
      }
      const unsigned vo = (unsigned)(wave * 4096 + lane * 16);
      u32x4 d[PP][4];
#define PLOAD(slot, n_) do { const unsigned o_ = (unsigned)(n_) * BK; \
        d[slot][0] = __builtin_amdgcn_raw_buffer_load_b128(rsX, voff[0], o_, 0); \
        d[slot][1] = __builtin_amdgcn_raw_buffer_load_b128(rsX, voff[1], o_, 0); \
        d[slot][2] = __builtin_amdgcn_raw_buffer_load_b128(rsX, voff[0], o_ + 16u * KT, 0); \
        d[slot][3] = __builtin_amdgcn_raw_buffer_load_b128(rsX, voff[1], o_ + 16u * KT, 0); } while (0)
      PLOAD(0, 0);
      __builtin_amdgcn_sched_barrier(0);
      CALL_STATE();
      __builtin_amdgcn_sched_barrier(0);
#pragma unroll
      for (int n = 1; n < PP; ++n) PLOAD(n, n);
      unsigned char* pws = p.ws[peer];
      const auto rsP = __builtin_amdgcn_make_buffer_rsrc(
          (void*)uni_ptr(pws + slot_off + (size_t)rank * IMG_SRC + (size_t)prb * IMG_RB), 0, (int)IMG_RB, 0x00020000);
      unsigned* pflag = (unsigned*)(pws + WS_FLAG + (size_t)((rank * NRB + prb) * 4 + wave) * 128);
      const unsigned long long tp0 = __builtin_amdgcn_s_memrealtime();
      for (int n0 = 0; n0 < NSTEP; n0 += PP) {
#pragma unroll
        for (int q = 0; q < PP; ++q) {
          const int n = n0 + q;
          while (__builtin_amdgcn_s_memrealtime() - tp0 < (unsigned long long)(n * PUSH_TICKS)) __builtin_amdgcn_s_sleep(1);
          const unsigned po = (unsigned)n * A_BYTES;
          __builtin_amdgcn_raw_buffer_store_b128(d[q][0], rsP, vo, po, AUX_SYS);
          __builtin_amdgcn_raw_buffer_store_b128(d[q][1], rsP, vo + 1024, po, AUX_SYS);
          __builtin_amdgcn_raw_buffer_store_b128(d[q][2], rsP, vo + 2048, po, AUX_SYS);
          __builtin_amdgcn_raw_buffer_store_b128(d[q][3], rsP, vo + 3072, po, AUX_SYS);
          WAIT_V(0);
          if (lane == 0) __hip_atomic_store(pflag, seq0 + (unsigned)(n + 1), __ATOMIC_RELAXED, __HIP_MEMORY_SCOPE_SYSTEM);
          if (n + PP < NSTEP) PLOAD(q, n + PP);
        }
      }
    }
    if (tid == 0) __hip_atomic_store(call_ctr, call_raw + 1u, __ATOMIC_RELAXED, __HIP_MEMORY_SCOPE_AGENT);
    return;
  }

  if (b < 4 * NR) {
    if (tid == 0) __hip_atomic_store(call_ctr, call_raw + 1u, __ATOMIC_RELAXED, __HIP_MEMORY_SCOPE_AGENT);
    return;
  }

  const int srel = 1 + ((idx - 4) >> 2);
  const int cb = (xcd & 1) * 4 + ((idx - 4) & 3);
  const int rb = xcd >> 1;
  const int src = (rank + srel) & (NR - 1);
  const int wm = wave >> 2, wn = wave & 3;
  const int sq = rb * (NR - 1) + (srel - 1);
  const int srowA = sq * 16, srowB = sq < 4 ? 448 + sq * 16 : srowA;
  const bool strip = wm == 0 || sq < 4;
  const float scale = __builtin_bit_cast(float, __builtin_amdgcn_readfirstlane(__builtin_bit_cast(int, p.sx[0] * p.sw[0])));

  unsigned xvoff[2], wvoff[2], svoff;
#pragma unroll
  for (int j = 0; j < 2; ++j) {
    xvoff[j] = (unsigned)(wave * 2048 + j * 1024 + lane * 16);
    const int k = wave * 16 + j * 4 + (lane >> 4);
    wvoff[j] = (unsigned)(k * NT + (((lane & 15) ^ ((k & 7) | (((k >> 4) & 1) << 3))) * 16));
  }
  {
    const int m = (wave & 3) * 4 + ((lane >> 3) & 3);
    svoff = (unsigned)(((wm ? srowB : srowA) + m) * KT + (((lane & 7) ^ ((m >> 1) & 7)) * 16));
  }
  CALL_STATE();
  const auto rsX = __builtin_amdgcn_make_buffer_rsrc(
      (void*)uni_ptr(ws + slot_off + (size_t)src * IMG_SRC + (size_t)rb * IMG_RB), 0, (int)IMG_RB, 0x00020000);
  const auto rsS = __builtin_amdgcn_make_buffer_rsrc((void*)uni_ptr(p.x), 0, MS * KT, 0x00020000);
  const auto rsW = __builtin_amdgcn_make_buffer_rsrc((void*)uni_ptr(p.w + (size_t)rank * NS + (size_t)cb * BN), 0,
                                                     (int)((size_t)KT * NT - (size_t)rank * NS - (size_t)cb * BN), 0x00020000);
  const unsigned* myflag = (const unsigned*)(ws + WS_FLAG + (size_t)((src * NRB + rb) * 4 + (wave >> 1)) * 128);

#define CLAMP(s) ((s) < NSTEP ? (s) : NSTEP - 1)
#define DMAX(so, s, k) __builtin_amdgcn_raw_ptr_buffer_load_lds(rsX, (LAS void*)(L + (so) + wave * 2048 + (k) * 1024), 16, \
    xvoff[k], (unsigned)CLAMP(s) * A_BYTES, 0, AUX_SYS)
#define DMAW(so, s, k) __builtin_amdgcn_raw_ptr_buffer_load_lds(rsW, (LAS void*)(L + (so) + A_BYTES + wave * 4096 + (k) * 1024), 16, \
    wvoff[(k) & 1], (unsigned)CLAMP(s) * (unsigned)(BK * NT) + (unsigned)((k) >> 1) * (8u * NT), 0, 0)
#define DMAS(so, s) do { if (lane < 32) __builtin_amdgcn_raw_ptr_buffer_load_lds(rsS, (LAS void*)(L + (so) + S_OFF + wave * 512), 16, \
    svoff, (unsigned)CLAMP(s) * BK, 0, 0); } while (0)
  auto load_x = [&](int s) { const unsigned so = (unsigned)((s % NSTAGE) * STAGE_BYTES); DMAX(so, s, 0); DMAX(so, s, 1); };
  auto load_w = [&](int s) { const unsigned so = (unsigned)((s % NSTAGE) * STAGE_BYTES);
    DMAW(so, s, 0); DMAW(so, s, 1); DMAW(so, s, 2); DMAW(so, s, 3); DMAS(so, s); };

  const int fr = lane & 15, g = lane >> 4;
  const unsigned xa0 = lbase + (unsigned)((wm * 64 + fr) * BK + ((g ^ ((fr >> 1) & 7)) * 16));
  const unsigned sa0 = lbase + (unsigned)(S_OFF + (wm * 16 + fr) * BK + ((g ^ ((fr >> 1) & 7)) * 16));
  unsigned wb[4];
  {
    const int q = fr >> 1, ph = fr & 1;
    const int f = q | ((g & 1) << 3);
#pragma unroll
    for (int ni = 0; ni < 4; ++ni)
      wb[ni] = lbase + (unsigned)(A_BYTES + (16 * g + q) * BN + (((wn * 4 + ni) ^ f) * 16) + ph * 8);
  }

  constexpr int ALL = NSTEP + 8;
  int known = 0;
  int xi = 0;
  int pend = 0;
#define UNI(v) v = __builtin_amdgcn_readfirstlane(v)
  LAS unsigned* actl = (LAS unsigned*)(L + NSTAGE * STAGE_BYTES + 32 + wave * 8);
  const unsigned actl_a = lbase + NSTAGE * STAGE_BYTES + 32 + wave * 8;

  auto pre_sync = [&](int j) {
    UNI(xi); UNI(pend);
    if (__builtin_expect(pend == 0 && xi == j + 2, 1)) { WAIT_V(7); return; }
    UNI(known);
    if (pend) {
      WAIT_V(4);
      unsigned pv;
      asm volatile("ds_read_b32 %0, %1\n\ts_waitcnt lgkmcnt(0)" : "=v"(pv) : "v"(actl_a) : "memory");
      const int k = (int)(__builtin_amdgcn_readfirstlane(pv) - seq0);
      known = k > known ? (k < NSTEP ? k : ALL) : known;
      pend = 0;
    } else if (xi > j) {
      WAIT_V(5);
    }
    if (xi <= j) {
      if (known <= j) {
        const unsigned v = wait_flag(myflag, seq0 + (unsigned)(j + 1), tmo, call);
        const int k = (int)(v - seq0);
        known = (v != 0xffffffffu && k < NSTEP) ? k : ALL;
      }
      load_x(j);
      xi = j + 1;
      if (known > j + 1) { load_x(j + 1); xi = j + 2; WAIT_V(2); } else WAIT_V(0);
    } else if (xi == j + 1 && known > j + 1) {
      load_x(j + 1);
      xi = j + 2;
    }
  };
  auto post_x = [&](int j) -> int {
    UNI(known); UNI(xi);
    if (__builtin_expect(xi == j + 2 && known > j + 2, 1)) { xi = j + 3; return 1; }
    return 0;
  };
  auto post_poll = [&](int j) {
    UNI(known);
    if (__builtin_expect(known < j + 6 || j == NSTEP - 1, 0)) {
      const int np = j < NSTEP - 1 ? 1 : 0;
      if (np && lane == 0) __builtin_amdgcn_global_load_lds(myflag, actl, 4, 0, AUX_SYS);
      if (!np && lane == 0) __builtin_amdgcn_global_load_lds(tmo, actl + 1, 4, 0, 16  );
      pend = np;
    }
  };

#define TR8(dst, addr, off) asm volatile("ds_read_b64_tr_b8 %0, %1 offset:%2" : "=v"(dst) : "v"(addr), "i"(off))
#define RD128(dst, addr, off) asm volatile("ds_read_b128 %0, %1 offset:%2" : "=v"(dst) : "v"(addr), "i"(off))
#define TRW(W, addr) do { const unsigned a_ = (addr); TR8(W##a, a_, 0); TR8(W##b, a_, 8 * BN); TR8(W##c, a_, 64 * BN); TR8(W##d, a_, 72 * BN); } while (0)
#define RDX(X, so_) do { const unsigned a_ = xa0 + (so_), b_ = (xa0 ^ 64u) + (so_); \
    RD128(X##0l, a_, 0); RD128(X##0h, b_, 0); RD128(X##1l, a_, 2048); RD128(X##1h, b_, 2048); \
    RD128(X##2l, a_, 4096); RD128(X##2h, b_, 4096); RD128(X##3l, a_, 6144); RD128(X##3h, b_, 6144); \
    RD128(X##4l, sa0 + (so_), 0); RD128(X##4h, (sa0 ^ 64u) + (so_), 0); } while (0)
#define CAT4(W) ((v8i){W##a[0], W##a[1], W##b[0], W##b[1], W##c[0], W##c[1], W##d[0], W##d[1]})
#define CAT2(l, h) ((v8i){l[0], l[1], l[2], l[3], h[0], h[1], h[2], h[3]})
#define MF1(ni, mi, Wv, Xv) asm volatile("v_mfma_f32_16x16x128_f8f6f4 %0, %1, %2, %0 cbsz:1" : "+a"(acc[ni][mi]) : "v"(Wv), "v"(Xv))
#define PINW1(n, W) asm volatile("s_waitcnt lgkmcnt(" #n ")" : "+v"(W##a), "+v"(W##b), "+v"(W##c), "+v"(W##d))
#define PINW2(W, V) asm volatile("s_waitcnt lgkmcnt(0)" : "+v"(W##a), "+v"(W##b), "+v"(W##c), "+v"(W##d), "+v"(V##a), "+v"(V##b), \
    "+v"(V##c), "+v"(V##d) :: "memory")
#define PINX(X, W) asm volatile("s_waitcnt lgkmcnt(0)" : "+v"(W##a), "+v"(W##b), "+v"(W##c), "+v"(W##d), "+v"(X##0l), "+v"(X##0h), \
    "+v"(X##1l), "+v"(X##1h), "+v"(X##2l), "+v"(X##2h), "+v"(X##3l), "+v"(X##3h), "+v"(X##4l), "+v"(X##4h))
#define SB0() __builtin_amdgcn_sched_barrier(0)
#define MFX(ni, mi, W, X) do { SB0(); MF1(ni, mi, CAT4(W), CAT2(X##mi##l, X##mi##h)); SB0(); } while (0)
#define MFS(ni, W, X) do { if (strip) MFX(ni, 4, W, X); } while (0)
#define KHALF1(XC, so_) \
    const unsigned a1_ = wb[1] + (so_), a2_ = wb[2] + (so_), a3_ = wb[3] + (so_); \
    MFX(0, 0, w0, XC); TR8(w1a, a1_, 0); TR8(w1b, a1_, 8 * BN); TR8(w1c, a1_, 64 * BN); \
    MFX(0, 1, w0, XC); TR8(w1d, a1_, 72 * BN); TR8(w2a, a2_, 0); TR8(w2b, a2_, 8 * BN); \
    MFX(0, 2, w0, XC); TR8(w2c, a2_, 64 * BN); TR8(w2d, a2_, 72 * BN); TR8(w3a, a3_, 0); \
    MFX(0, 3, w0, XC); TR8(w3b, a3_, 8 * BN); TR8(w3c, a3_, 64 * BN); TR8(w3d, a3_, 72 * BN); \
    MFS(0, w0, XC); \
    PINW1(8, w1);
#define KSTEP(i, XC, XN, so_, sn_, s2_) do { \
    KHALF1(XC, so_) \
    MFX(1, 0, w1, XC); DMAW(s2_, (i) + 2, 1); \
    MFX(1, 1, w1, XC); DMAW(s2_, (i) + 2, 2); \
    MFX(1, 2, w1, XC); DMAW(s2_, (i) + 2, 3); \
    MFX(1, 3, w1, XC); DMAS(s2_, (i) + 2); \
    MFS(1, w1, XC); \
    pre_sync((i) + 1); \
    PINW2(w2, w3); \
    BAR(); \
    const int nx_ = post_x((i) + 1); \
    const unsigned xl_ = xa0 + (sn_), xh_ = (xa0 ^ 64u) + (sn_), a0_ = wb[0] + (sn_); \
    MFX(2, 0, w2, XC); RD128(XN##0l, xl_, 0); RD128(XN##0h, xh_, 0); TR8(w0a, a0_, 0); \
    MFX(2, 1, w2, XC); TR8(w0b, a0_, 8 * BN); TR8(w0c, a0_, 64 * BN); TR8(w0d, a0_, 72 * BN); \
    MFX(2, 2, w2, XC); RD128(XN##1l, xl_, 2048); RD128(XN##1h, xh_, 2048); \
    MFX(2, 3, w2, XC); RD128(XN##2l, xl_, 4096); RD128(XN##2h, xh_, 4096); \
    MFS(2, w2, XC); \
    MFX(3, 0, w3, XC); RD128(XN##3l, xl_, 6144); RD128(XN##3h, xh_, 6144); \
    MFX(3, 1, w3, XC); RD128(XN##4l, sa0 + (sn_), 0); RD128(XN##4h, (sa0 ^ 64u) + (sn_), 0); \
    if (nx_) { DMAX(so_, (i) + 3, 0); DMAX(so_, (i) + 3, 1); } \
    MFX(3, 2, w3, XC); DMAW(so_, (i) + 3, 0); \
    MFX(3, 3, w3, XC); post_poll((i) + 1); \
    MFS(3, w3, XC); \
    PINX(XN, w0); SB0(); \
  } while (0)

  v4f acc[4][5];
  v2i w0a, w0b, w0c, w0d, w1a, w1b, w1c, w1d, w2a, w2b, w2c, w2d, w3a, w3b, w3c, w3d;
  v4i xa0l, xa0h, xa1l, xa1h, xa2l, xa2h, xa3l, xa3h, xa4l, xa4h, xb0l, xb0h, xb1l, xb1h, xb2l, xb2h, xb3l, xb3h, xb4l, xb4h;

  const auto rsO = __builtin_amdgcn_make_buffer_rsrc((void*)uni_ptr(p.out), 0, MT * NS * 4, 0x00020000);
  const unsigned ocol = (unsigned)(cb * BN + wn * 64 + fr * 4) * 4u;
  const unsigned ovo = ocol + (unsigned)(src * MS + rb * BM + wm * 64 + g) * (unsigned)(NS * 4);
  const unsigned svo = ocol + (unsigned)(rank * MS + (wm ? srowB : srowA) + g) * (unsigned)(NS * 4);
  load_w(0);
  load_w(1);
#pragma unroll
  for (int i = 0; i < 4; ++i)
#pragma unroll
    for (int j = 0; j < 5; ++j) acc[i][j] = (v4f){0.f, 0.f, 0.f, 0.f};
  pre_sync(0);
  BAR();
  if (post_x(0)) load_x(2);
  DMAW(2 * STAGE_BYTES, 2, 0);
  post_poll(0);
  RDX(xa, 0u);
  TRW(w0, wb[0]);
  PINX(xa, w0);
  SB0();
  {
    unsigned s0 = 0, s1 = STAGE_BYTES, s2 = 2 * STAGE_BYTES;
    for (int i = 0; i < NSTEP - 2; i += 2) {
      KSTEP(i, xa, xb, s0, s1, s2);
      KSTEP(i + 1, xb, xa, s1, s2, s0);
      const unsigned t_ = s0; s0 = s2; s2 = s1; s1 = t_;
    }
    KSTEP(NSTEP - 2, xa, xb, s0, s1, s2);
    constexpr float LOG2E = 1.4426950408889634f, LIM2 = 60.0f * LOG2E;
    const float nscale2 = -scale * LOG2E;
    const unsigned tbuf = lbase + s2 + (unsigned)(A_BYTES + wave * 4096);
    const unsigned twr = tbuf + (unsigned)(fr * 256), trd = tbuf + (unsigned)(g * 256);
    float oscale;
#define PINA(mi) asm volatile("" : "+a"(acc[0][mi]), "+a"(acc[1][mi]), "+a"(acc[2][mi]), "+a"(acc[3][mi]))
#define EPI1(ni, mi) do { \
      const v4f y = acc[ni][mi] * oscale; \
      const v4f z = acc[ni][mi] * nscale2; \
      v4f d; \
      _Pragma("unroll") for (int r = 0; r < 4; ++r) d[r] = __builtin_amdgcn_exp2f(__builtin_amdgcn_fmed3f(z[r], -LIM2, LIM2)); \
      d = d + 1.0f; \
      _Pragma("unroll") for (int r = 0; r < 4; ++r) d[r] = __builtin_amdgcn_rcpf(d[r]); \
      const v4f o = y * d; \
      asm volatile("ds_write_b128 %0, %1" :: "v"(twr + (unsigned)((((ni) * 4 + g) ^ fr) * 16)), "v"(o) : "memory"); \
    } while (0)
#define EPIT(t) asm volatile("ds_read_b128 %0, %1 offset:%2" : "=v"(o##t) : "v"(trd + (unsigned)((fr ^ ((t) * 4 + g)) * 16)), "i"((t) * 1024) : "memory")
#define EPI(mi) do { \
      EPI1(0, mi); EPI1(1, mi); EPI1(2, mi); EPI1(3, mi); \
      u32x4 o0, o1, o2, o3; \
      asm volatile("s_waitcnt lgkmcnt(0)" ::: "memory"); \
      EPIT(0); EPIT(1); EPIT(2); EPIT(3); \
      asm volatile("s_waitcnt lgkmcnt(0)" : "+v"(o0), "+v"(o1), "+v"(o2), "+v"(o3) :: "memory"); \
      const unsigned vo_ = (mi) < 4 ? ovo : svo, so_ = (mi) < 4 ? (unsigned)((mi) * 16 * NS * 4) : 0u; \
      __builtin_amdgcn_raw_buffer_store_b128(o0, rsO, vo_, so_, AUX_WT); \
      __builtin_amdgcn_raw_buffer_store_b128(o1, rsO, vo_, so_ + 4u * NS * 4, AUX_WT); \
      __builtin_amdgcn_raw_buffer_store_b128(o2, rsO, vo_, so_ + 8u * NS * 4, AUX_WT); \
      __builtin_amdgcn_raw_buffer_store_b128(o3, rsO, vo_, so_ + 12u * NS * 4, AUX_WT); \
    } while (0)
#define MF3(mi) do { MFX(1, mi, w1, xb); MFX(2, mi, w2, xb); MFX(3, mi, w3, xb); } while (0)
    {
      KHALF1(xb, s1)
      PINW2(w2, w3);
      MF3(0); MF3(1);
      unsigned tv;
      asm volatile("s_waitcnt vmcnt(0)\n\tds_read_b32 %0, %1 offset:4\n\ts_waitcnt lgkmcnt(0)" : "=v"(tv) : "v"(actl_a) : "memory");
      oscale = __builtin_amdgcn_readfirstlane(tv) == call ? __builtin_nanf("") : scale;
      MF3(2);
      PINA(0);
      EPI(0);
      MF3(3);
      PINA(1);
      EPI(1);
      if (strip) MF3(4);
      SB0();
      PINA(2);
      EPI(2);
      asm volatile("s_nop 15\n\ts_nop 15" ::: "memory");
      PINA(3);
      EPI(3);
      if (strip) { PINA(4); EPI(4); }
    }
  }
  if (tid == 0) __hip_atomic_store(call_ctr, call, __ATOMIC_RELAXED, __HIP_MEMORY_SCOPE_AGENT);
}

}

extern "C" void kernel_launch(int rank, int nranks, ncclComm_t comm,
                              hipStream_t stream,
                              void* const* d_in, const int* in_sizes, int n_in,
                              void* d_out, int out_size,
                              const int* in_dtypes, int out_dtype,
                              void* const* d_out_all,
                              void* const* d_ws_all, size_t ws_bytes) {
  (void)comm; (void)in_dtypes; (void)out_dtype; (void)d_out_all;
  if (nranks != NR || n_in < 4 || (unsigned)rank >= (unsigned)NR || ws_bytes < WS_TOTAL ||
      in_sizes[0] != MS * KT || in_sizes[1] != KT * NT || out_size != MT * NS) {
    fprintf(stderr, "unexpected problem shape\n");
    return;
  }
  Params p;
  p.x = (const unsigned char*)d_in[0];
  p.w = (const unsigned char*)d_in[1];
  p.sx = (const float*)d_in[2];
  p.sw = (const float*)d_in[3];
  p.out = (float*)d_out;
  for (int r = 0; r < NR; ++r) p.ws[r] = (unsigned char*)d_ws_all[r];
  p.rank = rank;
  hipLaunchKernelGGL(gemm_ag_kernel, dim3(NWG), dim3(512), 0, stream, p);
  HIP_CHECK(hipGetLastError());
}
```
